```python
import numpy as np

kernel_name = "gpu_a2a_gemm_m2048_k2048_n2048_bf16_gelu"


ABSMAX_RTOL = 0.008

COMPUTE_DTYPE = "bfloat16"
EPILOGUE = "gelu"
IN_QUANT = None
OUT_QUANT = None
IN_DTYPES = {"x": "bfloat16", "w_mat": "bfloat16"}
WORLD_SIZE = 8

def _to_bf16(x):
    u = np.ascontiguousarray(x, dtype=np.float32).view(np.uint32)
    return ((u + 0x7FFF + ((u >> 16) & 1)) & 0xFFFF0000).view(np.float32)

def setup_inputs():

    x = _to_bf16(np.random.randn(2048, 2048).astype(np.float32))
    w_mat = _to_bf16((np.random.randn(2048, 2048) / np.sqrt(2048)).astype(np.float32))
    return {"x": x, "w_mat": w_mat}

def _epilogue(y):
    c = 0.7978845608028654
    return (0.5 * y * (1.0 + np.tanh(c * (y + 0.044715 * y ** 3)))).astype(np.float32)

def reference(x, w_mat):
    y = (x @ w_mat).astype(np.float32)
    return _epilogue(y)
```

```cpp
#include <hip/hip_runtime.h>
#include <rccl/rccl.h>
#include <cstdio>
#include <cstdint>

#define HIP_CHECK(expr) do { hipError_t _e = (expr); if (_e != hipSuccess) { \
    fprintf(stderr, "HIP error %s at %s:%d\n", hipGetErrorString(_e), __FILE__, __LINE__); \
    return; } } while (0)

namespace {

constexpr int NR = 8;
constexpr int MLOC = 256, KDIM = 2048, NDIM = 2048, KLOC = 256;
constexpr int NWG = 256;
constexpr unsigned WS_CTR = 0;
constexpr unsigned WS_TMO = 2048;
constexpr unsigned WS_SLOT0 = 65536;
constexpr unsigned SLOT_BYTES = NR * MLOC * KLOC * 2;
constexpr unsigned NSLOT = 3;
constexpr size_t WS_NEED = (size_t)WS_SLOT0 + (size_t)NSLOT * SLOT_BYTES;
constexpr unsigned long long DEADLINE_TICKS = 100000000ull;

typedef __attribute__((ext_vector_type(4))) unsigned u32x4;
typedef __attribute__((ext_vector_type(2))) unsigned u32x2;
typedef __attribute__((ext_vector_type(4))) float f32x4;
typedef __attribute__((ext_vector_type(16))) float f32x16;
typedef __attribute__((ext_vector_type(8))) __bf16 bf16x8;
typedef __attribute__((ext_vector_type(2))) __bf16 bf16x2;
typedef __attribute__((address_space(3))) char lds_char;

struct Args {
  const char* x;
  const char* w;
  char* out;
  char* ws[NR];
  int rank;
};

constexpr int AUX_SYS = 17;

__device__ __forceinline__ __amdgpu_buffer_rsrc_t mk_rsrc(const void* p, unsigned bytes) {
  return __builtin_amdgcn_make_buffer_rsrc(const_cast<void*>(p), 0, bytes, 0x00020000);
}
__device__ __forceinline__ unsigned umin3(unsigned a, unsigned b, unsigned c) {
  return __builtin_elementwise_min(__builtin_elementwise_min(a, b), c);
}
__device__ __forceinline__ unsigned min16(const u32x4& a, const u32x4& b, const u32x4& c, const u32x4& d) {
  unsigned m0 = umin3(a[0], a[1], a[2]), m1 = umin3(a[3], b[0], b[1]);
  m0 = umin3(m0, b[2], b[3]); m1 = umin3(m1, c[0], c[1]);
  m0 = umin3(m0, c[2], c[3]); m1 = umin3(m1, d[0], d[1]);
  m0 = umin3(m0, d[2], d[3]);
  return __builtin_elementwise_min(m0, m1);
}
__device__ __forceinline__ float gelu_tanh(float y) {
  const float u = 0.7978845608028654f * (y + 0.044715f * y * y * y);
  const float e = __builtin_amdgcn_exp2f(-2.885390081777927f * u);
  return y * __builtin_amdgcn_rcpf(1.0f + e);
}

#define TR16(dst, addr, OFF) \
  asm volatile( \
    "ds_read_b64_tr_b16 %0, %16 offset:%c17+0\n\t"    "ds_read_b64_tr_b16 %1, %16 offset:%c17+512\n\t" \
    "ds_read_b64_tr_b16 %2, %18 offset:%c17+0\n\t"    "ds_read_b64_tr_b16 %3, %18 offset:%c17+512\n\t" \
    "ds_read_b64_tr_b16 %4, %16 offset:%c17+2048\n\t" "ds_read_b64_tr_b16 %5, %16 offset:%c17+2560\n\t" \
    "ds_read_b64_tr_b16 %6, %18 offset:%c17+2048\n\t" "ds_read_b64_tr_b16 %7, %18 offset:%c17+2560\n\t" \
    "ds_read_b64_tr_b16 %8, %16 offset:%c17+4096\n\t" "ds_read_b64_tr_b16 %9, %16 offset:%c17+4608\n\t" \
    "ds_read_b64_tr_b16 %10, %18 offset:%c17+4096\n\t" "ds_read_b64_tr_b16 %11, %18 offset:%c17+4608\n\t" \
    "ds_read_b64_tr_b16 %12, %16 offset:%c17+6144\n\t" "ds_read_b64_tr_b16 %13, %16 offset:%c17+6656\n\t" \
    "ds_read_b64_tr_b16 %14, %18 offset:%c17+6144\n\t" "ds_read_b64_tr_b16 %15, %18 offset:%c17+6656\n\t" \
    "s_waitcnt lgkmcnt(0)" \
    : "=&v"(dst[0]), "=&v"(dst[1]), "=&v"(dst[2]), "=&v"(dst[3]), "=&v"(dst[4]), "=&v"(dst[5]), "=&v"(dst[6]), "=&v"(dst[7]), \
      "=&v"(dst[8]), "=&v"(dst[9]), "=&v"(dst[10]), "=&v"(dst[11]), "=&v"(dst[12]), "=&v"(dst[13]), "=&v"(dst[14]), "=&v"(dst[15]) \
    : "v"(addr##0), "i"(OFF), "v"(addr##1) : "memory")

__device__ __forceinline__ bf16x8 mk8(const u32x2& lo, const u32x2& hi) {
  u32x4 v = {lo[0], lo[1], hi[0], hi[1]};
  return __builtin_bit_cast(bf16x8, v);
}

#define STAMP(i) do {} while (0)

__global__ __launch_bounds__(512, 2) void a2a_gemm_gelu(Args a) {
  __shared__ __attribute__((aligned(1024))) char lds[9 * 16384 + 64];
  STAMP(0);
  const unsigned lane = threadIdx.x & 63;
  const unsigned wv = __builtin_amdgcn_readfirstlane(threadIdx.x >> 6);
  const unsigned b = blockIdx.x;
  const unsigned rank = (unsigned)a.rank;
  const unsigned bn = (b & 7) * 4 + ((b >> 3) & 3), bm = b >> 5;
  char* const myws = a.ws[rank & 7];
  const bool own = (wv == rank);

  const __amdgpu_buffer_rsrc_t r_ws = mk_rsrc(myws, (unsigned)WS_NEED);
  const __amdgpu_buffer_rsrc_t r_x = mk_rsrc(a.x, 2048u * KLOC * 2);
  const unsigned frag_off = (lane & 31) * (KLOC * 2) + (lane >> 5) * 16;

  const unsigned reg_off = (wv + (wv > rank ? 1u : 0u)) * 16384u;
  lds_char* const lreg = (lds_char*)lds + reg_off;
  const char* const wsrc = a.w + (size_t)(wv * KLOC) * (NDIM * 2) + bn * 128;
  const unsigned wl_off = (lane >> 3) * (NDIM * 2) + (((lane & 7) ^ ((lane >> 2) & 4)) * 16);
  const unsigned trow = (lane >> 5) * 8 + ((lane & 15) >> 2);
  const unsigned tch = ((lane >> 4) & 1) * 2 + ((lane & 3) >> 1);
  const unsigned lbase = (unsigned)(size_t)lreg;
  const unsigned ta0 = lbase + trow * 128 + ((tch ^ ((trow & 2) << 1)) * 16) + (lane & 1) * 8;
  const unsigned ta1 = lbase + trow * 128 + (((tch + 4) ^ ((trow & 2) << 1)) * 16) + (lane & 1) * 8;

  u32x2 wf[64];
  u32x4 af[16];
  f32x16 acc0 = {0}, acc1 = {0};
  unsigned call;
  bool timed_out = false;

  if (own) {
    const unsigned dest = b & 7, sbm = (b >> 3) & 7, sq = b >> 6;
    const unsigned c0 = __builtin_amdgcn_raw_buffer_load_b32(r_ws, 0, WS_CTR + b * 4, 16);
    u32x4 sx[4];
    const unsigned s_src = (dest * MLOC + sbm * 32) * (KLOC * 2) + sq * 128;
#pragma unroll
    for (int i = 0; i < 4; ++i) sx[i] = __builtin_amdgcn_raw_buffer_load_b128(r_x, frag_off + i * 32, s_src, 0);
    const unsigned a_src = (rank * MLOC + bm * 32) * (KLOC * 2);
#pragma unroll
    for (int i = 0; i < 16; ++i) af[i] = __builtin_amdgcn_raw_buffer_load_b128(r_x, frag_off + i * 32, a_src, 0);
#pragma unroll
    for (int i = 0; i < 32; ++i)
      __builtin_amdgcn_global_load_lds((const void*)(wsrc + (size_t)i * 8 * (NDIM * 2) + wl_off), (__attribute__((address_space(3))) void*)(lreg + i * 1024), 16, 0, 0);
    call = __builtin_amdgcn_readfirstlane(c0) + 1;
    if (dest != rank && dest < (unsigned)NR) {
      const __amdgpu_buffer_rsrc_t r_dst = mk_rsrc(a.ws[dest], (unsigned)WS_NEED);
      const unsigned d_off = WS_SLOT0 + (call % NSLOT) * SLOT_BYTES + ((rank * 8 + sbm) * 16 + sq * 4) * 1024;
#pragma unroll
      for (int i = 0; i < 4; ++i) {
        u32x4 v = sx[i];
#pragma unroll
        for (int j = 0; j < 4; ++j) v[j] = v[j] ? v[j] : 0x8000u;
        __builtin_amdgcn_raw_buffer_store_b128(v, r_dst, lane * 16 + i * 1024, d_off, AUX_SYS);
      }
    }
    STAMP(1);
    {
      const unsigned z_off = WS_SLOT0 + ((call + 2) % NSLOT) * SLOT_BYTES + b * 4096;
      const u32x4 z = {0, 0, 0, 0};
#pragma unroll
      for (int i = 0; i < 4; ++i) __builtin_amdgcn_raw_buffer_store_b128(z, r_ws, lane * 16 + i * 1024, z_off, AUX_SYS);
    }
    if (dest != rank) asm volatile("s_waitcnt vmcnt(8)" ::: "memory");
    else asm volatile("s_waitcnt vmcnt(4)" ::: "memory");
    STAMP(2);
    TR16((wf + 0), ta, 0);
    TR16((wf + 16), ta, 8192);
    TR16((wf + 32), ta, 16384);
    TR16((wf + 48), ta, 24576);
    STAMP(3);
#pragma unroll
    for (int ks = 0; ks < 16; ++ks) {
      const bf16x8 av = __builtin_bit_cast(bf16x8, af[ks]);
      acc0 = __builtin_amdgcn_mfma_f32_32x32x16_bf16(av, mk8(wf[4 * ks], wf[4 * ks + 1]), acc0, 0, 0, 0);
      acc1 = __builtin_amdgcn_mfma_f32_32x32x16_bf16(av, mk8(wf[4 * ks + 2], wf[4 * ks + 3]), acc1, 0, 0, 0);
    }
    STAMP(4);
  } else {
    const unsigned c0 = __builtin_amdgcn_raw_buffer_load_b32(r_ws, 0, WS_CTR + b * 4, 16);
#pragma unroll
    for (int i = 0; i < 16; ++i)
      __builtin_amdgcn_global_load_lds((const void*)(wsrc + (size_t)i * 8 * (NDIM * 2) + wl_off), (__attribute__((address_space(3))) void*)(lreg + i * 1024), 16, 0, 0);
    asm volatile("s_waitcnt vmcnt(0)" ::: "memory");
    STAMP(1);
    TR16((wf + 0), ta, 0);
    TR16((wf + 16), ta, 8192);
#pragma unroll
    for (int i = 0; i < 16; ++i)
      __builtin_amdgcn_global_load_lds((const void*)(wsrc + (size_t)(i + 16) * 8 * (NDIM * 2) + wl_off), (__attribute__((address_space(3))) void*)(lreg + i * 1024), 16, 0, 0);
    call = __builtin_amdgcn_readfirstlane(c0) + 1;
    const unsigned a_off = WS_SLOT0 + (call % NSLOT) * SLOT_BYTES + (wv * 8 + bm) * 16384;
#pragma unroll
    for (int i = 0; i < 16; ++i) af[i] = __builtin_amdgcn_raw_buffer_load_b128(r_ws, lane * 16 + (i & 3) * 1024, a_off + (i >> 2) * 4096, AUX_SYS);
    asm volatile("s_waitcnt vmcnt(16)" ::: "memory");
    STAMP(2);
    TR16((wf + 32), ta, 0);
    TR16((wf + 48), ta, 8192);
    STAMP(3);
    unsigned long long t0 = 0;
#pragma unroll
    for (int g = 0; g < 4; ++g) {
      while (__builtin_amdgcn_ballot_w64(min16(af[4 * g], af[4 * g + 1], af[4 * g + 2], af[4 * g + 3]) == 0u) != 0ull) {
        const unsigned long long now = __builtin_amdgcn_s_memrealtime();
        if (t0 == 0) t0 = now;
        if (now - t0 > 100000ull) {
          const unsigned tm = __builtin_amdgcn_readfirstlane(
              __hip_atomic_load((unsigned*)(myws + WS_TMO), __ATOMIC_RELAXED, __HIP_MEMORY_SCOPE_AGENT));
          if (timed_out || tm == call || now - t0 > DEADLINE_TICKS) {
            __hip_atomic_store((unsigned*)(myws + WS_TMO), call, __ATOMIC_RELAXED, __HIP_MEMORY_SCOPE_AGENT);
            timed_out = true;
            break;
          }
          __builtin_amdgcn_s_sleep(32);
        }
        asm volatile("" ::: "memory");
#pragma unroll
        for (int i = 0; i < 4; ++i) af[4 * g + i] = __builtin_amdgcn_raw_buffer_load_b128(r_ws, lane * 16 + i * 1024, a_off + g * 4096, AUX_SYS);
      }
#pragma unroll
      for (int kk = 0; kk < 4; ++kk) {
        const int ks = 4 * g + kk;
        const bf16x8 av = __builtin_bit_cast(bf16x8, af[ks]);
        acc0 = __builtin_amdgcn_mfma_f32_32x32x16_bf16(av, mk8(wf[4 * ks], wf[4 * ks + 1]), acc0, 0, 0, 0);
        acc1 = __builtin_amdgcn_mfma_f32_32x32x16_bf16(av, mk8(wf[4 * ks + 2], wf[4 * ks + 3]), acc1, 0, 0, 0);
      }
    }
    STAMP(4);
    if (timed_out) acc0[0] = __builtin_nanf(""), acc0 = acc0 + acc0[0], acc1 = acc1 + acc0[0];
  }

  {
    const unsigned pa = lbase + lane * 4;
#pragma unroll
    for (int r = 0; r < 16; ++r) {
      asm volatile("ds_write_b32 %0, %1 offset:%c2" :: "v"(pa), "v"(acc0[r]), "i"(r * 256) : "memory");
      asm volatile("ds_write_b32 %0, %1 offset:%c2" :: "v"(pa), "v"(acc1[r]), "i"(4096 + r * 256) : "memory");
    }
    asm volatile("s_waitcnt lgkmcnt(0)\n\ts_barrier" ::: "memory");
  }
  STAMP(5);
  {
    const unsigned cblk = (lane >> 3) & 1, i = lane >> 5, h = (lane >> 4) & 1;
    const unsigned reg = 2 * wv + i;
    const unsigned ra = (unsigned)(size_t)lds + cblk * 4096 + reg * 256 + h * 128 + (lane & 7) * 16;
    f32x4 p[9];
    asm volatile(
        "ds_read_b128 %0, %9 offset:0\n\t"     "ds_read_b128 %1, %9 offset:16384\n\t"
        "ds_read_b128 %2, %9 offset:32768\n\t" "ds_read_b128 %3, %9 offset:49152\n\t"
        "ds_read_b128 %4, %10 offset:0\n\t"    "ds_read_b128 %5, %10 offset:16384\n\t"
        "ds_read_b128 %6, %10 offset:32768\n\t" "ds_read_b128 %7, %10 offset:49152\n\t"
        "ds_read_b128 %8, %11 offset:0\n\t"
        "s_waitcnt lgkmcnt(0)"
        : "=&v"(p[0]), "=&v"(p[1]), "=&v"(p[2]), "=&v"(p[3]), "=&v"(p[4]), "=&v"(p[5]), "=&v"(p[6]), "=&v"(p[7]), "=&v"(p[8])
        : "v"(ra), "v"(ra + 65536), "v"(ra + 131072) : "memory");
    f32x4 s = {0, 0, 0, 0};
#pragma unroll
    for (int r = 0; r < 9; ++r) {
      const bool skip = ((unsigned)r == rank + 1);
      s += skip ? f32x4{0, 0, 0, 0} : p[r];
    }
    const unsigned row = (reg & 3) + 8 * (reg >> 2) + 4 * h;
    bf16x2 o0 = {(__bf16)gelu_tanh(s[0]), (__bf16)gelu_tanh(s[1])};
    bf16x2 o1 = {(__bf16)gelu_tanh(s[2]), (__bf16)gelu_tanh(s[3])};
    u32x2 o = {__builtin_bit_cast(unsigned, o0), __builtin_bit_cast(unsigned, o1)};
    const __amdgpu_buffer_rsrc_t r_out = mk_rsrc(a.out, MLOC * NDIM * 2);
    __builtin_amdgcn_raw_buffer_store_b64(o, r_out, row * (NDIM * 2) + (lane & 15) * 8, (bm * 32) * (NDIM * 2) + bn * 128, 0);
    if (wv == 0 && lane == 0) __builtin_amdgcn_raw_buffer_store_b32(call, r_ws, 0, WS_CTR + b * 4, 16);
  }
}

}

extern "C" void kernel_launch(int rank, int nranks, ncclComm_t comm,
                              hipStream_t stream,
                              void* const* d_in, const int* in_sizes, int n_in,
                              void* d_out, int out_size,
                              const int* in_dtypes, int out_dtype,
                              void* const* d_out_all,
                              void* const* d_ws_all, size_t ws_bytes) {
  if (nranks != NR || (unsigned)rank >= (unsigned)NR || n_in < 2 || ws_bytes < WS_NEED ||
      in_sizes[0] != 2048 * KLOC || in_sizes[1] != KDIM * NDIM || out_size != MLOC * NDIM) {
    fprintf(stderr, "unexpected problem shape\n");
    return;
  }
  Args a;
  a.x = (const char*)d_in[0];
  a.w = (const char*)d_in[1];
  a.out = (char*)d_out;
  for (int p = 0; p < NR; ++p) a.ws[p] = (char*)d_ws_all[p];
  a.rank = rank;
  hipLaunchKernelGGL(a2a_gemm_gelu, dim3(NWG), dim3(512), 0, stream, a);
  HIP_CHECK(hipGetLastError());
}
```

```cpp
#include <hip/hip_runtime.h>
#include <rccl/rccl.h>
#include <cstdio>
#include <cstdint>

#define HIP_CHECK(expr) do { hipError_t _e = (expr); if (_e != hipSuccess) { \
    fprintf(stderr, "HIP error %s at %s:%d\n", hipGetErrorString(_e), __FILE__, __LINE__); \
    return; } } while (0)

#define PACE 0
#define WARM 1
#define DESTMAP 0
#define POLL_AT 4
#define OUT_AUX ""
#define NOSEND 0

namespace {

constexpr int NR = 8;
constexpr int MLOC = 256, KDIM = 2048, NDIM = 2048, KLOC = 256;
constexpr int NWG = 256;
constexpr unsigned WS_CTR = 0;
constexpr unsigned WS_TMO = 2048;
constexpr unsigned WS_SLOT0 = 65536;
constexpr unsigned SLOT_BYTES = NR * MLOC * KLOC * 2;
constexpr unsigned NSLOT = 4;
constexpr size_t WS_NEED = (size_t)WS_SLOT0 + (size_t)NSLOT * SLOT_BYTES;
constexpr unsigned long long DEADLINE_TICKS = 100000000ull;

typedef __attribute__((ext_vector_type(4))) unsigned u32x4;
typedef __attribute__((ext_vector_type(2))) unsigned u32x2;
typedef __attribute__((ext_vector_type(4))) float f32x4;
typedef __attribute__((ext_vector_type(16))) float f32x16;
typedef __attribute__((ext_vector_type(8))) __bf16 bf16x8;
typedef __attribute__((ext_vector_type(2))) __bf16 bf16x2;

typedef __attribute__((ext_vector_type(16))) unsigned u32x16;

struct Args {
  unsigned srd_x[4];
  unsigned srd_w[4];
  unsigned long long myws;
  unsigned long long out;
  unsigned rank, pad[3];
  unsigned srd_dst[NR][4];
};
__device__ __forceinline__ unsigned umin3(unsigned a, unsigned b, unsigned c) {
  return __builtin_elementwise_min(__builtin_elementwise_min(a, b), c);
}
__device__ __forceinline__ unsigned min16(const u32x4& a, const u32x4& b, const u32x4& c, const u32x4& d) {
  unsigned m0 = umin3(a[0], a[1], a[2]), m1 = umin3(a[3], b[0], b[1]);
  m0 = umin3(m0, b[2], b[3]); m1 = umin3(m1, c[0], c[1]);
  m0 = umin3(m0, c[2], c[3]); m1 = umin3(m1, d[0], d[1]);
  m0 = umin3(m0, d[2], d[3]);
  return __builtin_elementwise_min(m0, m1);
}
__device__ __forceinline__ float gelu_tanh(float y) {
  const float u = 0.7978845608028654f * (y + 0.044715f * y * y * y);
  const float e = __builtin_amdgcn_exp2f(-2.885390081777927f * u);
  return y * __builtin_amdgcn_rcpf(1.0f + e);
}
__device__ __forceinline__ bf16x8 mk8(const u32x2& lo, const u32x2& hi) {
  u32x4 v = {lo[0], lo[1], hi[0], hi[1]};
  return __builtin_bit_cast(bf16x8, v);
}

#define DMA4(M0V, V0, V1, V2, V3, SRD, SO) \
  asm volatile("s_mov_b32 m0, %0\n\ts_nop 0\n\t" \
               "buffer_load_dwordx4 %1, %5, %6 offen lds\n\t" \
               "buffer_load_dwordx4 %2, %5, %6 offen offset:1024 lds\n\t" \
               "buffer_load_dwordx4 %3, %5, %6 offen offset:2048 lds\n\t" \
               "buffer_load_dwordx4 %4, %5, %6 offen offset:3072 lds" \
               :: "s"(M0V), "v"(V0), "v"(V1), "v"(V2), "v"(V3), "s"(SRD), "s"(SO) : "memory")

#define TR8(VM, dst, OFF) \
  asm volatile( \
    "s_waitcnt vmcnt(%c11)\n\t" \
    "ds_read_b64_tr_b16 %0, %8 offset:%c10+0\n\t"    "ds_read_b64_tr_b16 %1, %8 offset:%c10+512\n\t" \
    "ds_read_b64_tr_b16 %2, %9 offset:%c10+0\n\t"    "ds_read_b64_tr_b16 %3, %9 offset:%c10+512\n\t" \
    "ds_read_b64_tr_b16 %4, %8 offset:%c10+2048\n\t" "ds_read_b64_tr_b16 %5, %8 offset:%c10+2560\n\t" \
    "ds_read_b64_tr_b16 %6, %9 offset:%c10+2048\n\t" "ds_read_b64_tr_b16 %7, %9 offset:%c10+2560\n\t" \
    "s_waitcnt lgkmcnt(0)" \
    : "=&v"((dst)[0]), "=&v"((dst)[1]), "=&v"((dst)[2]), "=&v"((dst)[3]), "=&v"((dst)[4]), "=&v"((dst)[5]), "=&v"((dst)[6]), "=&v"((dst)[7]) \
    : "v"(ta0), "v"(ta1), "i"(OFF), "i"(VM) : "memory")

#define PIECE(g, i) (4 * (g) + (i))
#define AF_OPS(g) "+v"(af[PIECE(g, 0)]), "+v"(af[PIECE(g, 1)]), "+v"(af[PIECE(g, 2)]), "+v"(af[PIECE(g, 3)])
#define AQ_OPS(g) [l16] "v"(lane16), [srd] "s"(srd_ws), \
    [q0] "s"(aq[PIECE(g, 0) >> 2]), [q1] "s"(aq[PIECE(g, 1) >> 2]), [q2] "s"(aq[PIECE(g, 2) >> 2]), [q3] "s"(aq[PIECE(g, 3) >> 2]), \
    [o0] "i"((PIECE(g, 0) & 3) * 1024), [o1] "i"((PIECE(g, 1) & 3) * 1024), [o2] "i"((PIECE(g, 2) & 3) * 1024), [o3] "i"((PIECE(g, 3) & 3) * 1024)
#define LOADG_TXT \
    "buffer_load_dwordx4 %0, %[l16], %[srd], %[q0] offen offset:%c[o0] sc0 sc1\n\t" \
    "buffer_load_dwordx4 %1, %[l16], %[srd], %[q1] offen offset:%c[o1] sc0 sc1\n\t" \
    "buffer_load_dwordx4 %2, %[l16], %[srd], %[q2] offen offset:%c[o2] sc0 sc1\n\t" \
    "buffer_load_dwordx4 %3, %[l16], %[srd], %[q3] offen offset:%c[o3] sc0 sc1\n\t"
#define LOADG(g) asm volatile(LOADG_TXT \
    : "=&v"(af[PIECE(g, 0)]), "=&v"(af[PIECE(g, 1)]), "=&v"(af[PIECE(g, 2)]), "=&v"(af[PIECE(g, 3)]) : AQ_OPS(g) : "memory")
#define WAITG(VM, g) asm volatile("s_waitcnt vmcnt(%c4)" : AF_OPS(g) : "i"(VM) : "memory")
#define MING(g) min16(af[PIECE(g, 0)], af[PIECE(g, 1)], af[PIECE(g, 2)], af[PIECE(g, 3)])
#define DECIDE(g, m) asm volatile( \
    "v_cmp_eq_u32_e32 vcc, 0, %[mn]\n\t" \
    "s_cbranch_vccz 1f\n\t" \
    LOADG_TXT \
    "s_branch 2f\n" \
    "1:\n\t" \
    "s_bitset0_b32 %[pend], %c[bit]\n" \
    "2:" \
    : AF_OPS(g), [pend] "+s"(pend) : [mn] "v"(m), [bit] "i"(g), AQ_OPS(g) : "memory", "vcc")
#define MFMA_KS(ks) do { \
    const bf16x8 av_ = __builtin_bit_cast(bf16x8, af[ks]); \
    acc0 = __builtin_amdgcn_mfma_f32_32x32x16_bf16(av_, mk8(wf[4 * (ks)], wf[4 * (ks) + 1]), acc0, 0, 0, 0); \
    acc1 = __builtin_amdgcn_mfma_f32_32x32x16_bf16(av_, mk8(wf[4 * (ks) + 2], wf[4 * (ks) + 3]), acc1, 0, 0, 0); } while (0)
#define MFMAG(g) do { MFMA_KS(PIECE(g, 0)); MFMA_KS(PIECE(g, 1)); MFMA_KS(PIECE(g, 2)); MFMA_KS(PIECE(g, 3)); } while (0)

#define STAMP(i) do {} while (0)

__global__ __launch_bounds__(512, 2) void a2a_gemm_gelu(Args a) {
  __shared__ __attribute__((aligned(1024))) char lds[160 * 1024];
  STAMP(0);
  const unsigned b = blockIdx.x;
  const unsigned dest = (b >> 3) & 7, sbm = b & 7, sq = b >> 6;
  u32x16 ka;
  u32x4 srd_dst;
  {
    const void* kp = (const void*)__builtin_amdgcn_kernarg_segment_ptr();
    asm volatile("s_load_dwordx16 %0, %2, 0x0\n\ts_load_dwordx4 %1, %2, %3"
                 : "=&s"(ka), "=&s"(srd_dst) : "s"(kp), "s"(0x40u + dest * 16u) : "memory");
  }
  const unsigned lane = threadIdx.x & 63;
  const unsigned wv = __builtin_amdgcn_readfirstlane(threadIdx.x >> 6);
  const unsigned bn = (b & 7) * 4 + ((b >> 3) & 3), bm = b >> 5;
  const unsigned lane16 = lane * 16;
  const unsigned ctr_off = WS_CTR + ((b & 7) * 32 + (b >> 3)) * 4;
  const unsigned frag_off = (lane & 31) * (KLOC * 2) + (lane >> 5) * 16;
  const unsigned s_src = (dest * MLOC + sbm * 32) * (KLOC * 2) + sq * 128;

  const unsigned lds0 = (unsigned)(size_t)(__attribute__((address_space(3))) char*)lds;
  const unsigned lring = lds0 + (wv ? 32768u + wv * 16384u : 0u);
  const unsigned wl0 = (lane >> 3) * (NDIM * 2) + (((lane & 7) ^ ((lane >> 2) & 4)) * 16);
  const unsigned wl1 = wl0 + 31744u, wl2 = wl0 + 2 * 31744u, wl3 = wl0 + 3 * 31744u;
  const unsigned trow = (lane >> 5) * 8 + ((lane & 15) >> 2);
  const unsigned tch = ((lane >> 4) & 1) * 2 + ((lane & 3) >> 1);
  const unsigned ta0 = lring + trow * 128 + ((tch ^ ((trow & 2) << 1)) * 16) + (lane & 1) * 8;
  const unsigned ta1 = lring + trow * 128 + (((tch + 4) ^ ((trow & 2) << 1)) * 16) + (lane & 1) * 8;

  asm volatile("s_waitcnt lgkmcnt(0)" : "+s"(ka), "+s"(srd_dst)
               : "v"(frag_off), "s"(s_src), "s"(ctr_off), "v"(wl0), "v"(wl1), "v"(wl2), "v"(wl3), "s"(lring) : "memory");
  const u32x4 srd_x = {ka[0], ka[1], ka[2], ka[3]}, srd_w = {ka[4], ka[5], ka[6], ka[7]};
  const u32x2 myws2 = {ka[8], ka[9]};
  char* const myws = (char*)(((unsigned long long)ka[9] << 32) | ka[8]);
  const unsigned rank = ka[12] & 7;
  const unsigned src = (rank + wv) & 7;
  const unsigned wso = src * (KLOC * NDIM * 2) + bn * 128;
  const u32x4 srd_ws = {ka[8], ka[9], (unsigned)WS_NEED, 0x00020000u};

  u32x2 wf[64];
  u32x4 af[16];
  f32x16 acc0 = {0}, acc1 = {0};
  unsigned call, c0;

  if (__builtin_expect(wv == 0, 1)) {
    u32x4 sx[4];
    asm volatile("s_setprio 3\n\t"
                 "buffer_load_dwordx4 %0, %5, %6, %7 offen\n\t"
                 "s_load_dword %4, %8, %9\n\t"
                 "buffer_load_dwordx4 %1, %5, %6, %7 offen offset:32\n\t"
                 "buffer_load_dwordx4 %2, %5, %6, %7 offen offset:64\n\t"
                 "buffer_load_dwordx4 %3, %5, %6, %7 offen offset:96"
                 : "=&v"(sx[0]), "=&v"(sx[1]), "=&v"(sx[2]), "=&v"(sx[3]), "=&s"(c0)
                 : "v"(frag_off), "s"(srd_x), "s"(s_src), "s"(myws2), "s"(ctr_off) : "memory");
    const unsigned d_unit = (sbm * 16 + sq * 4) * 1024;
    unsigned dummy;
    {
      const unsigned wo = (lane & 3) * SLOT_BYTES;
      asm volatile("buffer_load_dword %0, %1, %2, %3 offen sc0 sc1" : "=&v"(dummy) : "v"(wo), "s"(srd_dst), "s"(d_unit) : "memory");
    }
    {
      const unsigned hi = lane >> 5, ch = lane & 31;
      const unsigned aso = (rank * MLOC + bm * 32) * (KLOC * 2);
#pragma unroll
      for (int q = 0; q < 4; ++q) {
        unsigned v[4];
#pragma unroll
        for (int j = 0; j < 4; ++j) v[j] = hi * 512 + ((ch ^ (((q * 8 + j * 2) & 15) + hi)) * 16);
        DMA4(lds0 + 32768u + q * 4096u, v[0], v[1], v[2], v[3], srd_x, aso + q * 4096u);
      }
    }
#pragma unroll
    for (int q = 0; q < 8; ++q) DMA4(lds0 + q * 4096u, wl0, wl1, wl2, wl3, srd_w, wso + q * 131072u);
    asm volatile("s_waitcnt vmcnt(%c5) lgkmcnt(0)" : "+v"(sx[0]), "+v"(sx[1]), "+v"(sx[2]), "+v"(sx[3]), "+s"(c0) : "i"(48 + WARM) : "memory");
    STAMP(1);
    call = c0 + 1;
    {
      const unsigned d_off = d_unit + (call & (NSLOT - 1)) * SLOT_BYTES;
#pragma unroll
      for (int i = 0; i < 4; ++i) {
        u32x4 v = sx[i];
#pragma unroll
        for (int j = 0; j < 4; ++j) v[j] = v[j] ? v[j] : 0x8000u;
        asm volatile("buffer_store_dwordx4 %0, %1, %2, %3 offen offset:%c4 sc0 sc1\n\ts_nop 1"
                     :: "v"(v), "v"(lane16), "s"(srd_dst), "s"(d_off), "i"(i * 1024) : "memory");
      }
    }
    __builtin_amdgcn_s_setprio(0);
    STAMP(2);
    {
      const unsigned z_off = WS_SLOT0 + ((call + 2) & (NSLOT - 1)) * SLOT_BYTES + b * 4096;
      const u32x4 z = {0, 0, 0, 0};
      asm volatile("buffer_store_dwordx4 %0, %1, %2, %3 offen sc0 sc1\n\t"
                   "buffer_store_dwordx4 %0, %1, %2, %3 offen offset:1024 sc0 sc1\n\t"
                   "buffer_store_dwordx4 %0, %1, %2, %3 offen offset:2048 sc0 sc1\n\t"
                   "buffer_store_dwordx4 %0, %1, %2, %3 offen offset:3072 sc0 sc1\n\ts_nop 1"
                   :: "v"(z), "v"(lane16), "s"(srd_ws), "s"(z_off) : "memory");
    }
    {
      const unsigned r = lane & 31, h = lane >> 5;
      const unsigned ab = lds0 + 32768u + r * 512;
      unsigned aa[8];
#pragma unroll
      for (int j = 0; j < 8; ++j) aa[j] = ab + (((2 * j + h) ^ (r & 15)) * 16);
      asm volatile("s_waitcnt vmcnt(40)\n\t"
                   "ds_read_b128 %0, %16\n\t"  "ds_read_b128 %1, %17\n\t"  "ds_read_b128 %2, %18\n\t"  "ds_read_b128 %3, %19\n\t"
                   "ds_read_b128 %4, %20\n\t"  "ds_read_b128 %5, %21\n\t"  "ds_read_b128 %6, %22\n\t"  "ds_read_b128 %7, %23\n\t"
                   "ds_read_b128 %8, %16 offset:256\n\t"  "ds_read_b128 %9, %17 offset:256\n\t"  "ds_read_b128 %10, %18 offset:256\n\t" "ds_read_b128 %11, %19 offset:256\n\t"
                   "ds_read_b128 %12, %20 offset:256\n\t" "ds_read_b128 %13, %21 offset:256\n\t" "ds_read_b128 %14, %22 offset:256\n\t" "ds_read_b128 %15, %23 offset:256\n\t"
                   "s_waitcnt lgkmcnt(0)"
                   : "=&v"(af[0]), "=&v"(af[1]), "=&v"(af[2]), "=&v"(af[3]), "=&v"(af[4]), "=&v"(af[5]), "=&v"(af[6]), "=&v"(af[7]),
                     "=&v"(af[8]), "=&v"(af[9]), "=&v"(af[10]), "=&v"(af[11]), "=&v"(af[12]), "=&v"(af[13]), "=&v"(af[14]), "=&v"(af[15])
                   : "v"(aa[0]), "v"(aa[1]), "v"(aa[2]), "v"(aa[3]), "v"(aa[4]), "v"(aa[5]), "v"(aa[6]), "v"(aa[7]) : "memory");
    }
#pragma unroll
    for (int q = 0; q < 8; ++q) {
      TR8(36 - 4 * q, wf + 8 * q, q * 4096);
      MFMA_KS(2 * q); MFMA_KS(2 * q + 1);
    }
    asm volatile("" :: "v"(dummy));
    STAMP(3);
  } else {
#pragma unroll
    for (int q = 0; q < 4; ++q) DMA4(lring + q * 4096u, wl0, wl1, wl2, wl3, srd_w, wso + q * 131072u);
    asm volatile("s_load_dword %0, %1, %2" : "=&s"(c0) : "s"(myws2), "s"(ctr_off) : "memory");
#pragma unroll
    for (int q = 0; q < 4; ++q) {
      TR8(12, wf + 8 * q, q * 4096);
      DMA4(lring + q * 4096u, wl0, wl1, wl2, wl3, srd_w, wso + (q + 4) * 131072u);
      if (q == 0) STAMP(1);
    }
    asm volatile("" : "+s"(c0));
    call = c0 + 1;
    unsigned aq[4];
#pragma unroll
    for (int q = 0; q < 4; ++q) aq[q] = WS_SLOT0 + (call & (NSLOT - 1)) * SLOT_BYTES + (src * 8 + bm) * 16384 + q * 4096;
#pragma unroll
    for (int q = 4; q < 8; ++q) {
      if (q == POLL_AT) { LOADG(0); LOADG(1); LOADG(2); LOADG(3); }
      TR8((q >= POLL_AT ? 16 : 0) + 12 - 4 * (q - 4), wf + 8 * q, (q - 4) * 4096);
    }
    if (POLL_AT >= 8) { LOADG(0); LOADG(1); LOADG(2); LOADG(3); }
    STAMP(2);
    unsigned pend = 15;
#define FIRST(g) do { WAITG(12 - 4 * (g), g); const unsigned m_ = MING(g); DECIDE(g, m_); if (!(pend & (1u << (g)))) MFMAG(g); } while (0)
    FIRST(0); FIRST(1); FIRST(2); FIRST(3);
    STAMP(3);
    unsigned long long t0 = 0;
    while (pend) {
      const unsigned long long now = __builtin_amdgcn_s_memrealtime();
      if (t0 == 0) t0 = now;
      if (__builtin_expect(now - t0 > 100000ull, 0)) {
        const unsigned tm = __builtin_amdgcn_readfirstlane(
            __hip_atomic_load((unsigned*)(myws + WS_TMO), __ATOMIC_RELAXED, __HIP_MEMORY_SCOPE_AGENT));
        if (tm == call || now - t0 > DEADLINE_TICKS) {
          __hip_atomic_store((unsigned*)(myws + WS_TMO), call, __ATOMIC_RELAXED, __HIP_MEMORY_SCOPE_AGENT);
          break;
        }
        __builtin_amdgcn_s_sleep(32);
      }
      const unsigned was = pend;
      asm volatile("s_waitcnt vmcnt(0)" : AF_OPS(0), AF_OPS(1), AF_OPS(2), AF_OPS(3) :: "memory");
#define CHECK(g) do { if (was & (1u << (g))) { const unsigned m_ = MING(g); DECIDE(g, m_); } } while (0)
      CHECK(0); CHECK(1); CHECK(2); CHECK(3);
      const unsigned ok = was & ~pend;
      if (ok & 1) MFMAG(0);
      if (ok & 2) MFMAG(1);
      if (ok & 4) MFMAG(2);
      if (ok & 8) MFMAG(3);
    }
    if (__builtin_expect(pend != 0, 0)) {
      asm volatile("s_waitcnt vmcnt(0)" : AF_OPS(0), AF_OPS(1), AF_OPS(2), AF_OPS(3) :: "memory");
      acc0[0] = __builtin_nanf(""), acc0 = acc0 + acc0[0], acc1 = acc1 + acc0[0];
    }
    STAMP(4);
  }

  {
    const unsigned pa = lds0 + 32768u + wv * 16384u + lane * 4;
#pragma unroll
    for (int r = 0; r < 16; ++r) {
      *(volatile __attribute__((address_space(3))) float*)(size_t)(pa + r * 256) = acc0[r];
      *(volatile __attribute__((address_space(3))) float*)(size_t)(pa + 4096 + r * 256) = acc1[r];
    }
    asm volatile("s_waitcnt lgkmcnt(0)\n\ts_barrier" ::: "memory");
  }
  STAMP(5);
  {
    const unsigned cblk = (lane >> 3) & 1, i = lane >> 5, h = (lane >> 4) & 1;
    const unsigned reg = 2 * wv + i;
    const unsigned ra = lds0 + 32768u + cblk * 4096 + reg * 256 + h * 128 + (lane & 7) * 16;
    f32x4 p[8];
    asm volatile(
        "ds_read_b128 %0, %8 offset:0\n\t"     "ds_read_b128 %1, %8 offset:16384\n\t"
        "ds_read_b128 %2, %8 offset:32768\n\t" "ds_read_b128 %3, %8 offset:49152\n\t"
        "ds_read_b128 %4, %9 offset:0\n\t"     "ds_read_b128 %5, %9 offset:16384\n\t"
        "ds_read_b128 %6, %9 offset:32768\n\t" "ds_read_b128 %7, %9 offset:49152\n\t"
        "s_waitcnt lgkmcnt(0)"
        : "=&v"(p[0]), "=&v"(p[1]), "=&v"(p[2]), "=&v"(p[3]), "=&v"(p[4]), "=&v"(p[5]), "=&v"(p[6]), "=&v"(p[7])
        : "v"(ra), "v"(ra + 65536) : "memory");
    const f32x4 s = ((p[0] + p[1]) + (p[2] + p[3])) + ((p[4] + p[5]) + (p[6] + p[7]));
    const unsigned row = (reg & 3) + 8 * (reg >> 2) + 4 * h;
    bf16x2 o0 = {(__bf16)gelu_tanh(s[0]), (__bf16)gelu_tanh(s[1])};
    bf16x2 o1 = {(__bf16)gelu_tanh(s[2]), (__bf16)gelu_tanh(s[3])};
    u32x2 o = {__builtin_bit_cast(unsigned, o0), __builtin_bit_cast(unsigned, o1)};
    const u32x4 srd_out = {ka[10], ka[11], MLOC * NDIM * 2, 0x00020000u};
    const unsigned ovo = row * (NDIM * 2) + (lane & 15) * 8, oso = (bm * 32) * (NDIM * 2) + bn * 128;
    asm volatile("buffer_store_dwordx2 %0, %1, %2, %3 offen " OUT_AUX :: "v"(o), "v"(ovo), "s"(srd_out), "s"(oso) : "memory");
    if (wv == 0 && lane == 0) *(volatile unsigned*)(myws + ctr_off) = call;
  }
}

}

extern "C" void kernel_launch(int rank, int nranks, ncclComm_t comm,
                              hipStream_t stream,
                              void* const* d_in, const int* in_sizes, int n_in,
                              void* d_out, int out_size,
                              const int* in_dtypes, int out_dtype,
                              void* const* d_out_all,
                              void* const* d_ws_all, size_t ws_bytes) {
  if (nranks != NR || (unsigned)rank >= (unsigned)NR || n_in < 2 || ws_bytes < WS_NEED ||
      in_sizes[0] != 2048 * KLOC || in_sizes[1] != KDIM * NDIM || out_size != MLOC * NDIM) {
    fprintf(stderr, "unexpected problem shape\n");
    return;
  }
  auto srd = [](unsigned* d, const void* p, unsigned bytes) {
    const unsigned long long v = (unsigned long long)p;
    d[0] = (unsigned)v; d[1] = (unsigned)(v >> 32) & 0xffffu; d[2] = bytes; d[3] = 0x00020000u;
  };
  Args a = {};
  srd(a.srd_x, d_in[0], 2048u * KLOC * 2);
  srd(a.srd_w, d_in[1], KDIM * NDIM * 2);
  a.myws = (unsigned long long)d_ws_all[rank];
  a.out = (unsigned long long)d_out;
  a.rank = (unsigned)rank;
  const unsigned part = WS_SLOT0 + (unsigned)rank * (MLOC * KLOC * 2);
  for (int p = 0; p < NR; ++p)
    srd(a.srd_dst[p], (const char*)d_ws_all[p] + part, (p == rank || NOSEND) ? 0u : (unsigned)WS_NEED - part);
  hipLaunchKernelGGL(a2a_gemm_gelu, dim3(NWG), dim3(512), 0, stream, a);
  HIP_CHECK(hipGetLastError());
}
```

```cpp
#include <hip/hip_runtime.h>
#include <rccl/rccl.h>
#include <cstdio>
#include <cstdint>

#define HIP_CHECK(expr) do { hipError_t _e = (expr); if (_e != hipSuccess) { \
    fprintf(stderr, "HIP error %s at %s:%d\n", hipGetErrorString(_e), __FILE__, __LINE__); \
    return; } } while (0)

#define PACE 20
#define WARM 0
#define DESTMAP 0
#define POLL_AT 8
#define OUT_AUX ""
#define NOSEND 0
#define ROT 1
#define PRE 1
#define OWNA 1
#define PRIO 1
#define FAKEW 0
#define ZDELAY 0
#define ZWAIT 1

namespace {

constexpr int NR = 8;
constexpr int MLOC = 256, KDIM = 2048, NDIM = 2048, KLOC = 256;
constexpr int NWG = 256;
constexpr unsigned WS_CTR = 0;
constexpr unsigned WS_TMO = 2048;
constexpr unsigned WS_SLOT0 = 65536;
constexpr unsigned SLOT_BYTES = NR * MLOC * KLOC * 2;
constexpr unsigned NSLOT = 4;
constexpr unsigned RS = 20480;
constexpr size_t WS_NEED = (size_t)WS_SLOT0 + (size_t)NSLOT * SLOT_BYTES;
constexpr unsigned long long DEADLINE_TICKS = 100000000ull;

typedef __attribute__((ext_vector_type(4))) unsigned u32x4;
typedef __attribute__((ext_vector_type(2))) unsigned u32x2;
typedef __attribute__((ext_vector_type(4))) float f32x4;
typedef __attribute__((ext_vector_type(16))) float f32x16;
typedef __attribute__((ext_vector_type(8))) __bf16 bf16x8;
typedef __attribute__((ext_vector_type(2))) __bf16 bf16x2;

typedef __attribute__((ext_vector_type(16))) unsigned u32x16;

struct Args {
  unsigned srd_x[4];
  unsigned srd_w[4];
  unsigned long long myws;
  unsigned long long out;
  unsigned rank, pad[3];
  unsigned srd_dst[NR][4];
};
__device__ __forceinline__ unsigned umin3(unsigned a, unsigned b, unsigned c) {
  return __builtin_elementwise_min(__builtin_elementwise_min(a, b), c);
}
__device__ __forceinline__ unsigned min16(const u32x4& a, const u32x4& b, const u32x4& c, const u32x4& d) {
  unsigned m0 = umin3(a[0], a[1], a[2]), m1 = umin3(a[3], b[0], b[1]);
  m0 = umin3(m0, b[2], b[3]); m1 = umin3(m1, c[0], c[1]);
  m0 = umin3(m0, c[2], c[3]); m1 = umin3(m1, d[0], d[1]);
  m0 = umin3(m0, d[2], d[3]);
  return __builtin_elementwise_min(m0, m1);
}
__device__ __forceinline__ float gelu_tanh(float y) {
  const float u = 0.7978845608028654f * (y + 0.044715f * y * y * y);
  const float e = __builtin_amdgcn_exp2f(-2.885390081777927f * u);
  return y * __builtin_amdgcn_rcpf(1.0f + e);
}
__device__ __forceinline__ bf16x8 mk8(const u32x2& lo, const u32x2& hi) {
  u32x4 v = {lo[0], lo[1], hi[0], hi[1]};
  return __builtin_bit_cast(bf16x8, v);
}

#define DMA4(M0V, V0, V1, V2, V3, SRD, SO) \
  asm volatile("s_mov_b32 m0, %0\n\ts_nop 0\n\t" \
               "buffer_load_dwordx4 %1, %5, %6 offen lds\n\t" \
               "buffer_load_dwordx4 %2, %5, %6 offen offset:1024 lds\n\t" \
               "buffer_load_dwordx4 %3, %5, %6 offen offset:2048 lds\n\t" \
               "buffer_load_dwordx4 %4, %5, %6 offen offset:3072 lds" \
               :: "s"(M0V), "v"(V0), "v"(V1), "v"(V2), "v"(V3), "s"(SRD), "s"(SO) : "memory")

#define TR8(VM, dst, OFF) \
  asm volatile( \
    "s_waitcnt vmcnt(%c11)\n\t" \
    "ds_read_b64_tr_b16 %0, %8 offset:%c10+0\n\t"    "ds_read_b64_tr_b16 %1, %8 offset:%c10+512\n\t" \
    "ds_read_b64_tr_b16 %2, %9 offset:%c10+0\n\t"    "ds_read_b64_tr_b16 %3, %9 offset:%c10+512\n\t" \
    "ds_read_b64_tr_b16 %4, %8 offset:%c10+2048\n\t" "ds_read_b64_tr_b16 %5, %8 offset:%c10+2560\n\t" \
    "ds_read_b64_tr_b16 %6, %9 offset:%c10+2048\n\t" "ds_read_b64_tr_b16 %7, %9 offset:%c10+2560\n\t" \
    "s_waitcnt lgkmcnt(0)" \
    : "=&v"((dst)[0]), "=&v"((dst)[1]), "=&v"((dst)[2]), "=&v"((dst)[3]), "=&v"((dst)[4]), "=&v"((dst)[5]), "=&v"((dst)[6]), "=&v"((dst)[7]) \
    : "v"(ta0), "v"(ta1), "i"(OFF), "i"(VM) : "memory")

#define PIECE(g, i) (4 * (i) + (g))
#define QSEQ(i) (((((bm >> 1) + ((i) >> 1)) & 3) * 2) + (((i) ^ bm) & 1))
#define AF_OPS(g) "+v"(af[PIECE(g, 0)]), "+v"(af[PIECE(g, 1)]), "+v"(af[PIECE(g, 2)]), "+v"(af[PIECE(g, 3)])
#define AQ_OPS(g) [la] "v"(lane16), [srd] "s"(srd_ws), [q0] "s"(aq[g])
#define LOADG_TXT \
    "buffer_load_dwordx4 %0, %[la], %[srd], %[q0] offen sc0 sc1\n\t" \
    "buffer_load_dwordx4 %1, %[la], %[srd], %[q0] offen offset:1024 sc0 sc1\n\t" \
    "buffer_load_dwordx4 %2, %[la], %[srd], %[q0] offen offset:2048 sc0 sc1\n\t" \
    "buffer_load_dwordx4 %3, %[la], %[srd], %[q0] offen offset:3072 sc0 sc1\n\t"
#define LOADG(g) asm volatile(LOADG_TXT \
    : "=&v"(af[PIECE(g, 0)]), "=&v"(af[PIECE(g, 1)]), "=&v"(af[PIECE(g, 2)]), "=&v"(af[PIECE(g, 3)]) : AQ_OPS(g) : "memory")
#define WAITG(VM, g) asm volatile("s_waitcnt vmcnt(%c4)" : AF_OPS(g) : "i"(VM) : "memory")
#define MING(g) min16(af[PIECE(g, 0)], af[PIECE(g, 1)], af[PIECE(g, 2)], af[PIECE(g, 3)])
#define DECIDE(g, m) asm volatile( \
    "v_cmp_eq_u32_e32 vcc, 0, %[mn]\n\t" \
    "s_cbranch_vccz 1f\n\t" \
    LOADG_TXT \
    "s_branch 2f\n" \
    "1:\n\t" \
    "s_bitset0_b32 %[pend], %c[bit]\n" \
    "2:" \
    : AF_OPS(g), [pend] "+s"(pend) : [mn] "v"(m), [bit] "i"(g), AQ_OPS(g) : "memory", "vcc")
#define MFMA_KS(ks) do { \
    const bf16x8 av_ = __builtin_bit_cast(bf16x8, af[ks]); \
    acc0 = __builtin_amdgcn_mfma_f32_32x32x16_bf16(av_, mk8(wf[4 * (ks)], wf[4 * (ks) + 1]), acc0, 0, 0, 0); \
    acc1 = __builtin_amdgcn_mfma_f32_32x32x16_bf16(av_, mk8(wf[4 * (ks) + 2], wf[4 * (ks) + 3]), acc1, 0, 0, 0); } while (0)
#define MFMAG(g) do { MFMA_KS(PIECE(g, 0)); MFMA_KS(PIECE(g, 1)); MFMA_KS(PIECE(g, 2)); MFMA_KS(PIECE(g, 3)); } while (0)

#define STAMP(i) do {} while (0)

__global__ __launch_bounds__(512, 2) void a2a_gemm_gelu(Args a) {
  __shared__ __attribute__((aligned(1024))) char lds[160 * 1024];
  STAMP(0);
  const unsigned b = blockIdx.x;
  const unsigned dest = (b >> 3) & 7, sbm = b & 7, sq = b >> 6;
  u32x16 ka;
  u32x4 srd_dst;
  {
    const void* kp = (const void*)__builtin_amdgcn_kernarg_segment_ptr();
    asm volatile("s_load_dwordx16 %0, %2, 0x0\n\ts_load_dwordx4 %1, %2, %3"
                 : "=&s"(ka), "=&s"(srd_dst) : "s"(kp), "s"(0x40u + dest * 16u) : "memory");
  }
  const unsigned lane = threadIdx.x & 63;
  const unsigned wv = __builtin_amdgcn_readfirstlane(threadIdx.x >> 6);
  const unsigned bn = (b & 7) * 4 + ((b >> 3) & 3), bm = b >> 5;
  const unsigned lane16 = lane * 16;
  const unsigned ctr_off = WS_CTR + ((b & 7) * 32 + (b >> 3)) * 4;
  const unsigned frag_off = (lane & 31) * (KLOC * 2) + (lane >> 5) * 16;
  const unsigned s_src = (dest * MLOC + sbm * 32) * (KLOC * 2) + sq * 128 + ROT * (sbm & 1) * 64, s_src2 = s_src ^ 64;

  const unsigned lds0 = (unsigned)(size_t)(__attribute__((address_space(3))) char*)lds;
  const unsigned lring = lds0 + wv * RS;
  const unsigned wl0 = (lane >> 3) * (NDIM * 2) + (((lane & 7) ^ ((lane >> 2) & 4)) * 16);
  const unsigned wl1 = wl0 + 31744u, wl2 = wl0 + 2 * 31744u, wl3 = wl0 + 3 * 31744u;
#define WQS 131072u
#define WBASE(s) ((s) * (KLOC * NDIM * 2) + bn * 128)
  const unsigned trow = (lane >> 5) * 8 + ((lane & 15) >> 2);
  const unsigned tch = ((lane >> 4) & 1) * 2 + ((lane & 3) >> 1);
  const unsigned ta0 = lring + trow * 128 + ((tch ^ ((trow & 2) << 1)) * 16) + (lane & 1) * 8;
  const unsigned ta1 = lring + trow * 128 + (((tch + 4) ^ ((trow & 2) << 1)) * 16) + (lane & 1) * 8;

  asm volatile("s_waitcnt lgkmcnt(0)" : "+s"(ka), "+s"(srd_dst)
               : "v"(frag_off), "s"(s_src), "s"(s_src2), "s"(ctr_off), "v"(wl0), "v"(wl1), "v"(wl2), "v"(wl3), "s"(lring) : "memory");
  const u32x4 srd_x = {ka[0], ka[1], ka[2], ka[3]}, srd_w = {ka[4], ka[5], ka[6], ka[7]};
  const u32x2 myws2 = {ka[8], ka[9]};
  char* const myws = (char*)(((unsigned long long)ka[9] << 32) | ka[8]);
  const unsigned rank = ka[12] & 7;
  const unsigned src = (rank + wv) & 7;
  const unsigned wso = WBASE(src);
  const u32x4 srd_ws = {ka[8], ka[9], (unsigned)WS_NEED, 0x00020000u};

  const unsigned oq = (bm + wv) & 7;
  const unsigned own_w = WBASE(rank) + oq * WQS;
  const unsigned own_a = (rank * MLOC + bm * 32) * (KLOC * 2) + oq * 64;

  u32x2 wf[64];
  u32x4 af[16];
  u32x2 ow[8];
  u32x4 oa[2];
  f32x16 acc0 = {0}, acc1 = {0};
  unsigned call, c0;
  const unsigned oav = (lane >> 2) * (KLOC * 2) + (lane & 3) * 16;
  const unsigned oar = lring + 16384u + (lane & 31) * 64 + (lane >> 5) * 16;
#define LOAD_OA() asm volatile("s_mov_b32 m0, %0\n\ts_nop 0\n\t" \
    "buffer_load_dwordx4 %1, %3, %4 offen lds\n\tbuffer_load_dwordx4 %2, %3, %4 offen offset:1024 lds" \
    :: "s"(lring + 16384u), "v"(oav), "v"(oav + 7168u), "s"(srd_x), "s"(own_a) : "memory")
#define READ_OA() asm volatile("ds_read_b128 %0, %2\n\tds_read_b128 %1, %2 offset:32\n\ts_waitcnt lgkmcnt(0)" \
    : "=&v"(oa[0]), "=&v"(oa[1]) : "v"(oar) : "memory")
#define MFMA_OWN() do { asm volatile("" : "+v"(oa[0]), "+v"(oa[1])); \
    acc0 = __builtin_amdgcn_mfma_f32_32x32x16_bf16(__builtin_bit_cast(bf16x8, oa[0]), mk8(ow[0], ow[1]), acc0, 0, 0, 0); \
    acc1 = __builtin_amdgcn_mfma_f32_32x32x16_bf16(__builtin_bit_cast(bf16x8, oa[0]), mk8(ow[2], ow[3]), acc1, 0, 0, 0); \
    acc0 = __builtin_amdgcn_mfma_f32_32x32x16_bf16(__builtin_bit_cast(bf16x8, oa[1]), mk8(ow[4], ow[5]), acc0, 0, 0, 0); \
    acc1 = __builtin_amdgcn_mfma_f32_32x32x16_bf16(__builtin_bit_cast(bf16x8, oa[1]), mk8(ow[6], ow[7]), acc1, 0, 0, 0); } while (0)
#define DMAW(slot, so) DMA4(lring + (slot) * 4096u, wl0, wl1, wl2, wl3, srd_w, so)
#define WSO(i) (wso + QSEQ(i) * WQS)

  if (__builtin_expect(wv == 0, 1)) {
    u32x4 sx[4];
    asm volatile("s_setprio 3\n\t"
                 "buffer_load_dwordx4 %0, %5, %6, %7 offen\n\t"
                 "s_load_dword %4, %8, %9\n\t"
                 "buffer_load_dwordx4 %1, %5, %6, %7 offen offset:32\n\t"
                 "buffer_load_dwordx4 %2, %5, %6, %10 offen\n\t"
                 "buffer_load_dwordx4 %3, %5, %6, %10 offen offset:32"
                 : "=&v"(sx[0]), "=&v"(sx[1]), "=&v"(sx[2]), "=&v"(sx[3]), "=&s"(c0)
                 : "v"(frag_off), "s"(srd_x), "s"(s_src), "s"(myws2), "s"(ctr_off), "s"(s_src2) : "memory");
    const unsigned d_unit = sbm * 16384 + ((sq - ROT * (sbm >> 1)) & 3) * 1024;
    DMAW(0, own_w);
    LOAD_OA();
    asm volatile("s_waitcnt lgkmcnt(0)" : "+s"(c0) :: "memory");
    call = c0 + 1;
    const unsigned d_off = d_unit + (call & (NSLOT - 1)) * SLOT_BYTES;
#pragma unroll
    for (int i = 0; i < 4; ++i) {
      asm volatile("s_waitcnt vmcnt(%c1)" : "+v"(sx[i]) : "i"(9 + WARM), "s"(d_off) : "memory");
      if (i == 0) STAMP(1);
      u32x4 v = sx[i];
#pragma unroll
      for (int j = 0; j < 4; ++j) v[j] = v[j] ? v[j] : 0x8000u;
      asm volatile("buffer_store_dwordx4 %0, %1, %2, %3 offen sc0 sc1\n\ts_nop 1"
                   :: "v"(v), "v"(lane16), "s"(srd_dst), "s"(d_off + i * 4096) : "memory");
      if (i == (PACE ? 0 : 3)) asm volatile("s_barrier" ::: "memory");
      if (i < 3) __builtin_amdgcn_s_sleep(PACE);
    }
    __builtin_amdgcn_s_setprio(0);
    STAMP(2);
    TR8(4 + WARM, ow, 0);
    READ_OA();
    MFMA_OWN();
    STAMP(3);
    asm volatile("s_waitcnt vmcnt(0)" ::: "memory");
    STAMP(4);
    {
      const unsigned z_off = WS_SLOT0 + ((call + 2) & (NSLOT - 1)) * SLOT_BYTES + b * 4096;
      const u32x4 z = {0, 0, 0, 0};
      asm volatile("buffer_store_dwordx4 %0, %1, %2, %3 offen sc0 sc1\n\t"
                   "buffer_store_dwordx4 %0, %1, %2, %3 offen offset:1024 sc0 sc1\n\t"
                   "buffer_store_dwordx4 %0, %1, %2, %3 offen offset:2048 sc0 sc1\n\t"
                   "buffer_store_dwordx4 %0, %1, %2, %3 offen offset:3072 sc0 sc1\n\ts_nop 1"
                   :: "v"(z), "v"(lane16), "s"(srd_ws), "s"(z_off) : "memory");
    }
  } else {
    __builtin_amdgcn_s_setprio(2);
    DMAW(0, WSO(0));
    asm volatile("s_barrier" ::: "memory");
    LOAD_OA();
    asm volatile("s_load_dword %0, %1, %2" : "=&s"(c0) : "s"(myws2), "s"(ctr_off) : "memory");
    DMAW(1, WSO(1)); DMAW(2, own_w); DMAW(3, WSO(2));
    __builtin_amdgcn_s_setprio(1);
    TR8(14, wf + 0, 0);         DMAW(0, WSO(3));
    STAMP(1);
    TR8(12, wf + 8, 4096);      DMAW(1, WSO(4));
    TR8(12, ow, 8192);          DMAW(2, WSO(5));
    READ_OA();
    MFMA_OWN();
    TR8(12, wf + 16, 12288);    DMAW(3, WSO(6));
    TR8(12, wf + 24, 0);        DMAW(0, WSO(7));
#define Q0 4
#define RINGN 4
    __builtin_amdgcn_s_setprio(0);
    asm volatile("" : "+s"(c0));
    call = c0 + 1;
    unsigned aq[4];
#pragma unroll
    for (int g = 0; g < 4; ++g) aq[g] = WS_SLOT0 + (call & (NSLOT - 1)) * SLOT_BYTES + (src * 8 + bm) * 16384 + g * 4096;
#pragma unroll
    for (int q = Q0; q < 8; ++q) {
      if (q == POLL_AT) { LOADG(0); LOADG(1); LOADG(2); LOADG(3); }
      TR8((q >= POLL_AT ? 16 : 0) + 4 * (7 - q), wf + 8 * q, ((q + 1) % RINGN) * 4096);
    }
    if (POLL_AT >= 8) { LOADG(0); LOADG(1); LOADG(2); LOADG(3); }
    STAMP(2);
    unsigned pend = 15;
#define FIRST(g) do { WAITG(12 - 4 * (g), g); const unsigned m_ = MING(g); DECIDE(g, m_); if (!(pend & (1u << (g)))) MFMAG(g); } while (0)
    FIRST(0); FIRST(1); FIRST(2); FIRST(3);
    STAMP(3);
    unsigned long long t0 = 0;
    while (pend) {
      const unsigned long long now = __builtin_amdgcn_s_memrealtime();
      if (t0 == 0) t0 = now;
      if (__builtin_expect(now - t0 > 100000ull, 0)) {
        const unsigned tm = __builtin_amdgcn_readfirstlane(
            __hip_atomic_load((unsigned*)(myws + WS_TMO), __ATOMIC_RELAXED, __HIP_MEMORY_SCOPE_AGENT));
        if (tm == call || now - t0 > DEADLINE_TICKS) {
          __hip_atomic_store((unsigned*)(myws + WS_TMO), call, __ATOMIC_RELAXED, __HIP_MEMORY_SCOPE_AGENT);
          break;
        }
        __builtin_amdgcn_s_sleep(32);
      }
      const unsigned was = pend;
      asm volatile("s_waitcnt vmcnt(0)" : AF_OPS(0), AF_OPS(1), AF_OPS(2), AF_OPS(3) :: "memory");
#define CHECK(g) do { if (was & (1u << (g))) { const unsigned m_ = MING(g); DECIDE(g, m_); } } while (0)
      CHECK(0); CHECK(1); CHECK(2); CHECK(3);
      const unsigned ok = was & ~pend;
      if (ok & 1) MFMAG(0);
      if (ok & 2) MFMAG(1);
      if (ok & 4) MFMAG(2);
      if (ok & 8) MFMAG(3);
    }
    if (__builtin_expect(pend != 0, 0)) {
      asm volatile("s_waitcnt vmcnt(0)" : AF_OPS(0), AF_OPS(1), AF_OPS(2), AF_OPS(3) :: "memory");
      acc0[0] = __builtin_nanf(""), acc0 = acc0 + acc0[0], acc1 = acc1 + acc0[0];
    }
    STAMP(4);
  }

  {
    const unsigned pa = lring + lane * 4;
#pragma unroll
    for (int r = 0; r < 16; ++r) {
      *(volatile __attribute__((address_space(3))) float*)(size_t)(pa + r * 256) = acc0[r];
      *(volatile __attribute__((address_space(3))) float*)(size_t)(pa + 4096 + r * 256) = acc1[r];
    }
    asm volatile("s_waitcnt lgkmcnt(0)\n\ts_barrier" ::: "memory");
  }
  STAMP(5);
  {
    const unsigned cblk = (lane >> 3) & 1, i = lane >> 5, h = (lane >> 4) & 1;
    const unsigned reg = 2 * wv + i;
    const unsigned ra = lds0 + cblk * 4096 + reg * 256 + h * 128 + (lane & 7) * 16;
    f32x4 p[8];
    asm volatile(
        "ds_read_b128 %0, %8 offset:0\n\t"     "ds_read_b128 %1, %8 offset:20480\n\t"
        "ds_read_b128 %2, %8 offset:40960\n\t" "ds_read_b128 %3, %8 offset:61440\n\t"
        "ds_read_b128 %4, %9 offset:0\n\t"     "ds_read_b128 %5, %9 offset:20480\n\t"
        "ds_read_b128 %6, %9 offset:40960\n\t" "ds_read_b128 %7, %9 offset:61440\n\t"
        "s_waitcnt lgkmcnt(0)"
        : "=&v"(p[0]), "=&v"(p[1]), "=&v"(p[2]), "=&v"(p[3]), "=&v"(p[4]), "=&v"(p[5]), "=&v"(p[6]), "=&v"(p[7])
        : "v"(ra), "v"(ra + 4 * RS) : "memory");
    const f32x4 s = ((p[0] + p[1]) + (p[2] + p[3])) + ((p[4] + p[5]) + (p[6] + p[7]));
    const unsigned row = (reg & 3) + 8 * (reg >> 2) + 4 * h;
    bf16x2 o0 = {(__bf16)gelu_tanh(s[0]), (__bf16)gelu_tanh(s[1])};
    bf16x2 o1 = {(__bf16)gelu_tanh(s[2]), (__bf16)gelu_tanh(s[3])};
    u32x2 o = {__builtin_bit_cast(unsigned, o0), __builtin_bit_cast(unsigned, o1)};
    const u32x4 srd_out = {ka[10], ka[11], MLOC * NDIM * 2, 0x00020000u};
    const unsigned ovo = row * (NDIM * 2) + (lane & 15) * 8, oso = (bm * 32) * (NDIM * 2) + bn * 128;
    asm volatile("buffer_store_dwordx2 %0, %1, %2, %3 offen " OUT_AUX :: "v"(o), "v"(ovo), "s"(srd_out), "s"(oso) : "memory");
    if (wv == 0 && lane == 0) *(volatile unsigned*)(myws + ctr_off) = call;
  }
}

}

extern "C" void kernel_launch(int rank, int nranks, ncclComm_t comm,
                              hipStream_t stream,
                              void* const* d_in, const int* in_sizes, int n_in,
                              void* d_out, int out_size,
                              const int* in_dtypes, int out_dtype,
                              void* const* d_out_all,
                              void* const* d_ws_all, size_t ws_bytes) {
  if (nranks != NR || (unsigned)rank >= (unsigned)NR || n_in < 2 || ws_bytes < WS_NEED ||
      in_sizes[0] != 2048 * KLOC || in_sizes[1] != KDIM * NDIM || out_size != MLOC * NDIM) {
    fprintf(stderr, "unexpected problem shape\n");
    return;
  }
  auto srd = [](unsigned* d, const void* p, unsigned bytes) {
    const unsigned long long v = (unsigned long long)p;
    d[0] = (unsigned)v; d[1] = (unsigned)(v >> 32) & 0xffffu; d[2] = bytes; d[3] = 0x00020000u;
  };
  Args a = {};
  srd(a.srd_x, d_in[0], 2048u * KLOC * 2);
  srd(a.srd_w, d_in[1], KDIM * NDIM * 2);
  a.myws = (unsigned long long)d_ws_all[rank];
  a.out = (unsigned long long)d_out;
  a.rank = (unsigned)rank;
  const unsigned part = WS_SLOT0 + (unsigned)rank * (MLOC * KLOC * 2);
  for (int p = 0; p < NR; ++p)
    srd(a.srd_dst[p], (const char*)d_ws_all[p] + part, (p == rank || NOSEND) ? 0u : (unsigned)WS_NEED - part);
  hipLaunchKernelGGL(a2a_gemm_gelu, dim3(NWG), dim3(512), 0, stream, a);
  HIP_CHECK(hipGetLastError());
}
```

```cpp
#include <hip/hip_runtime.h>
#include <rccl/rccl.h>
#include <cstdio>
#include <cstdint>

#define HIP_CHECK(expr) do { hipError_t _e = (expr); if (_e != hipSuccess) { \
    fprintf(stderr, "HIP error %s at %s:%d\n", hipGetErrorString(_e), __FILE__, __LINE__); \
    return; } } while (0)

#define X0 70
#define DQ 32
#define NH 6
#define GAP 0
#define TINY 0
#define OUT_AUX ""
#define NOSEND 0
#define NOCHECK (NOSEND || TINY)
#define WORD 000,010,001,002, 003,004,005,006, 007,020,011,012, 013,014,015,016, 017,030,021,022, 023,024,025,026, 027,031,032,033, 034,035,036,037
#define ORD 0,1,2,3,8,4,5,6,7,10,11,12,13
#define DEPTH 4
#define PRE 1
#define PRIO 1
#define ZMODE 3
#define ZAUX "sc0 sc1"
#define OWNT 0
#define RED 0
#define NAT 1
#define XF 0
#define CTRM 1
#define ALATE 0

namespace {

constexpr int NR = 8;
constexpr int MLOC = 256, KDIM = 2048, NDIM = 2048, KLOC = 256;
constexpr int NWG = 256;
constexpr unsigned WS_CTR = 0;
constexpr unsigned WS_TMO = 2048;
constexpr unsigned WS_SLOT0 = 65536;
constexpr unsigned SLOT_BYTES = NR * MLOC * KLOC * 2;
constexpr unsigned NSLOT = 4;
constexpr unsigned WBIAS = 4096;
constexpr unsigned RS = 20480;
constexpr size_t WS_NEED = (size_t)WS_SLOT0 + (size_t)NSLOT * SLOT_BYTES;
#define SLOWT 100000ull
#define DLT 100000000ull
constexpr unsigned long long DEADLINE_TICKS = DLT;

typedef __attribute__((ext_vector_type(4))) unsigned u32x4;
typedef __attribute__((ext_vector_type(2))) unsigned u32x2;
typedef __attribute__((ext_vector_type(4))) float f32x4;
typedef __attribute__((ext_vector_type(16))) float f32x16;
typedef __attribute__((ext_vector_type(8))) __bf16 bf16x8;
typedef __attribute__((ext_vector_type(2))) __bf16 bf16x2;

typedef __attribute__((ext_vector_type(16))) unsigned u32x16;

struct Args {
  unsigned srd_x[4];
  unsigned srd_w[4];
  unsigned long long myws;
  unsigned long long out;
  unsigned rank, pad[3];
  unsigned srd_dst[NR][4];
  unsigned srd_wv[NR][4];
};
__device__ __forceinline__ unsigned umin3(unsigned a, unsigned b, unsigned c) {
  return __builtin_elementwise_min(__builtin_elementwise_min(a, b), c);
}
__device__ __forceinline__ unsigned min16(const u32x4& a, const u32x4& b, const u32x4& c, const u32x4& d) {
  unsigned m0 = umin3(a[0], a[1], a[2]), m1 = umin3(a[3], b[0], b[1]);
  m0 = umin3(m0, b[2], b[3]); m1 = umin3(m1, c[0], c[1]);
  m0 = umin3(m0, c[2], c[3]); m1 = umin3(m1, d[0], d[1]);
  m0 = umin3(m0, d[2], d[3]);
  return __builtin_elementwise_min(m0, m1);
}
__device__ __forceinline__ float gelu_tanh(float y) {
  const float t = __builtin_fmaf(y * y, -0.10294324f, -2.30220819f);
  const float e = __builtin_amdgcn_exp2f(y * t);
  return y * __builtin_amdgcn_rcpf(1.0f + e);
}
__device__ __forceinline__ bf16x8 mk8(const u32x2& lo, const u32x2& hi) {
  u32x4 v = {lo[0], lo[1], hi[0], hi[1]};
  return __builtin_bit_cast(bf16x8, v);
}

#define DMA4(M0V, V, SRD, S0, S1, S2, S3) \
  asm volatile("s_mov_b32 m0, %0\n\ts_nop 0\n\t" \
               "buffer_load_dwordx4 %1, %2, %3 offen lds\n\t" \
               "buffer_load_dwordx4 %1, %2, %4 offen offset:1024 lds\n\t" \
               "buffer_load_dwordx4 %1, %2, %5 offen offset:2048 lds\n\t" \
               "buffer_load_dwordx4 %1, %2, %6 offen offset:3072 lds" \
               :: "s"(M0V), "v"(V), "s"(SRD), "s"(S0), "s"(S1), "s"(S2), "s"(S3) : "memory")

#define TR8(VM, dst, OFF) \
  asm volatile( \
    "s_waitcnt vmcnt(%c11)\n\t" \
    "ds_read_b64_tr_b16 %0, %8 offset:%c10+0\n\t"    "ds_read_b64_tr_b16 %1, %8 offset:%c10+512\n\t" \
    "ds_read_b64_tr_b16 %2, %9 offset:%c10+0\n\t"    "ds_read_b64_tr_b16 %3, %9 offset:%c10+512\n\t" \
    "ds_read_b64_tr_b16 %4, %8 offset:%c10+2048\n\t" "ds_read_b64_tr_b16 %5, %8 offset:%c10+2560\n\t" \
    "ds_read_b64_tr_b16 %6, %9 offset:%c10+2048\n\t" "ds_read_b64_tr_b16 %7, %9 offset:%c10+2560\n\t" \
    "s_waitcnt lgkmcnt(0)" \
    : "=&v"((dst)[0]), "=&v"((dst)[1]), "=&v"((dst)[2]), "=&v"((dst)[3]), "=&v"((dst)[4]), "=&v"((dst)[5]), "=&v"((dst)[6]), "=&v"((dst)[7]) \
    : "v"(ta0), "v"(ta1), "i"(OFF), "i"(VM) : "memory")

#define PIECE(g, i) (4 * (g) + (i))
constexpr unsigned WSEQ[32] = {WORD};
constexpr bool wseq_ok() { unsigned m = 0; for (int i = 0; i < 32; ++i) m |= 1u << (WSEQ[i] & 31); return m == 0xffffffffu; }
static_assert(wseq_ok(), "WORD must be a permutation");
constexpr unsigned stab(int i, int h) { unsigned v = 0; for (int g = 0; g < 4; ++g) v |= WSEQ[8 * g + 2 * i + h] << (8 * g); return v; }
constexpr int SEQ[13] = {ORD};
constexpr int NIT = 13;
constexpr int item_cnt(int t) { return t >= NIT ? 0 : SEQ[t] == 8 ? 6 : 4; }
constexpr int ring_slot(int t) { int n = 0; for (int i = 0; i < t && i < NIT; ++i) n += SEQ[i] <= 8; return n % 4; }
constexpr int last_w() { int l = 0; for (int i = 0; i < NIT; ++i) if (SEQ[i] <= 8) l = i; return l; }
constexpr int issued_at(int t) { int n = t + DEPTH; if (n > NIT) n = NIT; if (ALATE && t <= last_w() && n > last_w() + 1) n = last_w() + 1; if (ALATE && t > last_w()) n = NIT; return n; }
constexpr int vm_at(int t) { int n = 0; for (int i = t + 1; i < issued_at(t); ++i) n += item_cnt(i); return n; }
#define AF_OPS(g) "+v"(af[PIECE(g, 0)]), "+v"(af[PIECE(g, 1)]), "+v"(af[PIECE(g, 2)]), "+v"(af[PIECE(g, 3)])
#define AQ_OPS(g) [la] "v"(lane16), [srd] "s"(srd_ws), [q0] "s"(aq[g])
#define LOADG_TXT \
    "buffer_load_dwordx4 %0, %[la], %[srd], %[q0] offen sc0 sc1\n\t" \
    "buffer_load_dwordx4 %1, %[la], %[srd], %[q0] offen offset:1024 sc0 sc1\n\t" \
    "buffer_load_dwordx4 %2, %[la], %[srd], %[q0] offen offset:2048 sc0 sc1\n\t" \
    "buffer_load_dwordx4 %3, %[la], %[srd], %[q0] offen offset:3072 sc0 sc1\n\t"
#define LOADG(g) asm volatile(LOADG_TXT \
    : "=&v"(af[PIECE(g, 0)]), "=&v"(af[PIECE(g, 1)]), "=&v"(af[PIECE(g, 2)]), "=&v"(af[PIECE(g, 3)]) : AQ_OPS(g) : "memory")
#define WAITG(VM, g) asm volatile("s_waitcnt vmcnt(%c4)" : AF_OPS(g) : "i"(VM) : "memory")
#define MING(g) min16(af[PIECE(g, 0)], af[PIECE(g, 1)], af[PIECE(g, 2)], af[PIECE(g, 3)])
#define DECIDE(g, m) asm volatile( \
    "v_cmp_eq_u32_e32 vcc, 0, %[mn]\n\t" \
    "s_cbranch_vccz 1f\n\t" \
    LOADG_TXT \
    "s_branch 2f\n" \
    "1:\n\t" \
    "s_bitset0_b32 %[pend], %c[bit]\n" \
    "2:" \
    : AF_OPS(g), [pend] "+s"(pend) : [mn] "v"(m), [bit] "i"(g), AQ_OPS(g) : "memory", "vcc")
#define MFMA_KS(ks) do { \
    const bf16x8 av_ = __builtin_bit_cast(bf16x8, af[ks]); \
    acc0 = __builtin_amdgcn_mfma_f32_32x32x16_bf16(av_, mk8(wf[4 * (ks)], wf[4 * (ks) + 1]), acc0, 0, 0, 0); \
    acc1 = __builtin_amdgcn_mfma_f32_32x32x16_bf16(av_, mk8(wf[4 * (ks) + 2], wf[4 * (ks) + 3]), acc1, 0, 0, 0); } while (0)
#define MFMAG(g) do { MFMA_KS(PIECE(g, 0)); MFMA_KS(PIECE(g, 1)); MFMA_KS(PIECE(g, 2)); MFMA_KS(PIECE(g, 3)); } while (0)

#define STAMP(i) do {} while (0)

__global__ __launch_bounds__(512, 2) void a2a_gemm_gelu(Args a) {
  __shared__ __attribute__((aligned(1024))) char lds[160 * 1024];
  STAMP(0);
  const unsigned b = blockIdx.x;
  const void* const kp = (const void*)__builtin_amdgcn_kernarg_segment_ptr();
  u32x16 ka;
  const unsigned wv = __builtin_amdgcn_readfirstlane(threadIdx.x >> 6);
  const unsigned lds0 = (unsigned)(size_t)(__attribute__((address_space(3))) char*)lds;

  unsigned lane = threadIdx.x & 63, blk = b;
#define FENCE_IDS() asm volatile("" : "+v"(lane), "+s"(blk))
#define bn (((blk & 7) * 4 + ((blk >> 3) & 3)))
#define bm (blk >> 5)
#define lane16 (lane * 16)
#define ctr_off (WS_CTR + ((blk & 7) * 32 + (blk >> 3)) * 4)
#define lring (lds0 + wv * RS)
#define WL0 ((lane >> 3) * (NDIM * 2) + (((lane & 7) ^ ((lane >> 2) & 4)) * 16))
#define trow ((lane >> 5) * 8 + ((lane & 15) >> 2))
#define tch (((lane >> 4) & 1) * 2 + ((lane & 3) >> 1))
#define LATE_IDS() \
  const unsigned ta0 = lring + trow * 128 + ((tch ^ ((trow & 2) << 1)) * 16) + (lane & 1) * 8; \
  const unsigned ta1 = lring + trow * 128 + (((tch + 4) ^ ((trow & 2) << 1)) * 16) + (lane & 1) * 8; \
  const unsigned oar = lring + 16384u + (lane & 31) * 64 + (lane >> 5) * 16
#define srd_x (u32x4{ka[0], ka[1], ka[2], ka[3]})
#define srd_w (u32x4{ka[4], ka[5], ka[6], ka[7]})
#define myws2 (u32x2{ka[8], ka[9]})
#define myws ((char*)(((unsigned long long)ka[9] << 32) | ka[8]))
#define rank (ka[12] & 7)
#define srd_ws (u32x4{ka[8], ka[9], (unsigned)WS_NEED, 0x00020000u})
#define oq ((bm + wv) & 7)
#define own_w (bn * 128 + WBIAS + oq * 131072u)
#define own_a ((rank * MLOC + bm * 32) * (KLOC * 2) + oq * 64)
#define oav ((lane >> 2) * (KLOC * 2) + (lane & 3) * 16)

  u32x2 wf[64];
  u32x4 af[16];
  u32x2 ow[8];
  u32x4 oa[2];
  f32x16 acc0 = {0}, acc1 = {0};
  unsigned call, c0;
#define LOAD_OA() asm volatile("s_mov_b32 m0, %0\n\ts_nop 0\n\t" \
    "buffer_load_dwordx4 %1, %3, %4 offen lds\n\tbuffer_load_dwordx4 %2, %3, %4 offen offset:1024 lds" \
    :: "s"(lring + 16384u), "v"(oav), "v"(oav + 7168u), "s"(srd_x), "s"(own_a) : "memory")
#define READ_OA() asm volatile("ds_read_b128 %0, %2\n\tds_read_b128 %1, %2 offset:32\n\ts_waitcnt lgkmcnt(0)" \
    : "=&v"(oa[0]), "=&v"(oa[1]) : "v"(oar) : "memory")
#define MFMA_OWN() do { asm volatile("" : "+v"(oa[0]), "+v"(oa[1])); \
    acc0 = __builtin_amdgcn_mfma_f32_32x32x16_bf16(__builtin_bit_cast(bf16x8, oa[0]), mk8(ow[0], ow[1]), acc0, 0, 0, 0); \
    acc1 = __builtin_amdgcn_mfma_f32_32x32x16_bf16(__builtin_bit_cast(bf16x8, oa[0]), mk8(ow[2], ow[3]), acc1, 0, 0, 0); \
    acc0 = __builtin_amdgcn_mfma_f32_32x32x16_bf16(__builtin_bit_cast(bf16x8, oa[1]), mk8(ow[4], ow[5]), acc0, 0, 0, 0); \
    acc1 = __builtin_amdgcn_mfma_f32_32x32x16_bf16(__builtin_bit_cast(bf16x8, oa[1]), mk8(ow[6], ow[7]), acc1, 0, 0, 0); } while (0)
#define DMA_OWN(slot) DMA4(lring + (slot) * 4096u, wl0, srd_w, own_w, own_w + 31744u, own_w + 2 * 31744u, own_w + 3 * 31744u)
#define WPOS(n) (((bm + ((n) >> 2)) & 7) * 131072u + bn * 128 + (WBIAS + ((n) & 3) * 31744u))
#define DMA_W(slot, it) DMA4(lring + (slot) * 4096u, wl0, srd_wv, WPOS(4 * (it)), WPOS(4 * (it) + 1), WPOS(4 * (it) + 2), WPOS(4 * (it) + 3))

#define ZERO4() do { \
      const unsigned z_off = WS_SLOT0 + ((call + 2) & (NSLOT - 1)) * SLOT_BYTES + b * 4096; \
      const u32x4 z = {0, 0, 0, 0}; \
      asm volatile("buffer_store_dwordx4 %0, %1, %2, %3 offen " ZAUX "\n\t" \
                   "buffer_store_dwordx4 %0, %1, %2, %3 offen offset:1024 " ZAUX "\n\t" \
                   "buffer_store_dwordx4 %0, %1, %2, %3 offen offset:2048 " ZAUX "\n\t" \
                   "buffer_store_dwordx4 %0, %1, %2, %3 offen offset:3072 " ZAUX "\n\ts_nop 1" \
                   :: "v"(z), "v"(lane16), "s"(srd_ws), "s"(z_off) : "memory"); } while (0)
  if (__builtin_expect(wv == 0, 1)) {
    const unsigned dest = (b >> 3) & 7, sbm = b & 7, sq = b >> 6;
    unsigned long long tstart;
    u32x4 srd_dst;
    asm volatile("s_load_dwordx16 %0, %1, 0x0\n\ts_setprio 3" : "=&s"(ka) : "s"(kp) : "memory");
    const unsigned xo = (lane & 31) * (KLOC * 2) + (lane >> 5) * 16;
    const unsigned s_src = (dest * MLOC + sbm * 32) * (KLOC * 2);
    const unsigned s0 = s_src + ((sbm + 2 * sq) & 7) * 64, s1 = s_src + ((sbm + 2 * sq + 1) & 7) * 64;
    asm volatile("s_waitcnt lgkmcnt(0)" : "+s"(ka) : "v"(xo), "s"(s0), "s"(s1) : "memory");
    u32x4 sx[4];
    asm volatile("buffer_load_dwordx4 %0, %4, %5, %6 offen\n\t"
                 "buffer_load_dwordx4 %1, %4, %5, %6 offen offset:32\n\t"
                 "buffer_load_dwordx4 %2, %4, %5, %7 offen\n\t"
                 "buffer_load_dwordx4 %3, %4, %5, %7 offen offset:32"
                 : "=&v"(sx[0]), "=&v"(sx[1]), "=&v"(sx[2]), "=&v"(sx[3])
                 : "v"(xo), "s"(srd_x), "s"(s0), "s"(s1) : "memory");
    FENCE_IDS();
    asm volatile("s_memrealtime %1\n\ts_load_dword %0, %3, %4\n\ts_load_dwordx4 %2, %5, %6"
                 : "=&s"(c0), "=&s"(tstart), "=&s"(srd_dst) : "s"(myws2), "s"(ctr_off), "s"(kp), "s"(0x40u + dest * 16u) : "memory");
    const unsigned wl0 = WL0;
    DMA_OWN(0);
    LOAD_OA();
    FENCE_IDS();
    const unsigned slot = sq * 8 + ((sbm + rank - dest) & 7);
    const bool hold = slot < NH && dest != rank;
    if (!hold) asm volatile("s_barrier" ::: "memory");
    LATE_IDS();
    asm volatile("s_waitcnt lgkmcnt(0)" : "+s"(c0), "+s"(tstart), "+s"(srd_dst) :: "memory");
    call = c0 + 1;
    const unsigned d_off = sbm * 16384 + sq * 4096 + (call & (NSLOT - 1)) * SLOT_BYTES;
    const unsigned sv = lane16;
    const unsigned tgt = (unsigned)tstart + X0 + ((slot * DQ) >> 2) + (slot < NH ? 0 : GAP);
    asm volatile("s_waitcnt vmcnt(%c4)" : "+v"(sx[0]), "+v"(sx[1]), "+v"(sx[2]), "+v"(sx[3])
                 : "i"(OWNT == 0 ? 6 : 0), "s"(d_off), "s"(tgt), "v"(sv), "v"(ta0), "v"(ta1), "v"(oar) : "memory");
    if (__builtin_expect(__builtin_amdgcn_ballot_w64(min16(sx[0], sx[1], sx[2], sx[3]) == 0) != 0, 0)) {
#pragma unroll
      for (int i = 0; i < 4; ++i)
#pragma unroll
        for (int j = 0; j < 4; ++j) sx[i][j] = sx[i][j] ? sx[i][j] : 0x8000u;
    }
    STAMP(1);
    asm volatile("" :: "v"(sx[0]), "v"(sx[1]), "v"(sx[2]), "v"(sx[3]));
    while ((int)((unsigned)__builtin_amdgcn_s_memrealtime() - tgt) < 0) {}
    asm volatile("buffer_store_dwordx4 %0, %4, %5, %6 offen sc0 sc1\n\t"
                 "buffer_store_dwordx4 %1, %4, %5, %6 offen offset:1024 sc0 sc1\n\t"
                 "buffer_store_dwordx4 %2, %4, %5, %6 offen offset:2048 sc0 sc1\n\t"
                 "buffer_store_dwordx4 %3, %4, %5, %6 offen offset:3072 sc0 sc1\n\ts_nop 1"
                 :: "v"(sx[0]), "v"(sx[1]), "v"(sx[2]), "v"(sx[3]), "v"(sv), "s"(srd_dst), "s"(d_off) : "memory");
    ZERO4();
#define ZN 4
    if (hold) asm volatile("s_barrier" ::: "memory");
    __builtin_amdgcn_s_setprio(0);
    STAMP(2);
    TR8(4 + ZN, ow, 0);
    READ_OA();
    MFMA_OWN();
    STAMP(3);
    STAMP(4);
  } else {
    unsigned aq[4];
    unsigned pend = 15;
#define ISSUE(t) do { if constexpr ((t) < NIT) { constexpr int it_ = SEQ[(t) < NIT ? (t) : 0]; \
      if constexpr (it_ < 8) DMA_W(ring_slot(t), it_ & 7); \
      else if constexpr (it_ == 8) { DMA_OWN(ring_slot(t)); LOAD_OA(); } \
      else LOADG((it_ - 10) & 3); } } while (0)
#define ISSUE_RANGE(lo, hi) do { \
      if constexpr ((lo) < (hi)) ISSUE(lo); if constexpr ((lo) + 1 < (hi)) ISSUE((lo) + 1); \
      if constexpr ((lo) + 2 < (hi)) ISSUE((lo) + 2); if constexpr ((lo) + 3 < (hi)) ISSUE((lo) + 3); } while (0)
#define STEP(t) do { constexpr int it_ = SEQ[t]; \
      if constexpr (it_ < 8) { TR8(vm_at(t), wf + 8 * (it_ & 7), ring_slot(t) * 4096); } \
      else if constexpr (it_ == 8) { TR8(vm_at(t), ow, ring_slot(t) * 4096); } \
      else { WAITG(vm_at(t), (it_ - 10) & 3); } \
      if constexpr ((t) == 0) STAMP(1); \
      if constexpr ((t) == 0) { \
        asm volatile("" : "+s"(c0)); call = c0 + 1; \
        for (int g = 0; g < 4; ++g) aq[g] = WS_SLOT0 + (call & (NSLOT - 1)) * SLOT_BYTES + (src * 8 + bm) * 16384 + g * 4096; } \
      if constexpr (PRIO && (t) == 5) __builtin_amdgcn_s_setprio(0); \
      ISSUE_RANGE(issued_at(t), issued_at((t) + 1)); \
      if constexpr ((t) == last_w()) STAMP(2); \
      if constexpr (it_ == 8) { READ_OA(); MFMA_OWN(); } \
      if constexpr (it_ >= 10) { const unsigned m_ = MING((it_ - 10) & 3); DECIDE((it_ - 10) & 3, m_); \
        if (!(pend & (1u << ((it_ - 10) & 3)))) MFMAG((it_ - 10) & 3); } } while (0)
    u32x4 srd_wv;
    asm volatile("s_load_dwordx4 %0, %2, %3\n\ts_load_dwordx16 %1, %2, 0x0" : "=&s"(srd_wv), "=&s"(ka) : "s"(kp), "s"(0xc0u + wv * 16u) : "memory");
    __builtin_amdgcn_s_setprio(2);
    const unsigned wl0 = WL0;
    asm volatile("s_waitcnt lgkmcnt(0)" : "+s"(ka), "+s"(srd_wv)
                 : "v"(wl0), "s"(lring), "s"(WPOS(0)), "s"(WPOS(1)), "s"(WPOS(2)), "s"(WPOS(3)) : "memory");
    ISSUE_RANGE(0, PRE);
    FENCE_IDS();
    asm volatile("s_barrier\n\ts_load_dword %0, %1, %2" : "=&s"(c0) : "s"(myws2), "s"(ctr_off) : "memory");
    ISSUE_RANGE(PRE, DEPTH);
    FENCE_IDS();
    LATE_IDS();
    const unsigned src = (rank + wv) & 7;
    __builtin_amdgcn_s_setprio(1);
    STEP(0); STEP(1); STEP(2); STEP(3); STEP(4); STEP(5); STEP(6); STEP(7); STEP(8); STEP(9); STEP(10); STEP(11); STEP(12);
    STAMP(3);
    unsigned long long t0 = 0;
    while (pend) {
      const unsigned long long now = __builtin_amdgcn_s_memrealtime();
      if (t0 == 0) t0 = now;
      if (__builtin_expect(now - t0 > SLOWT, 0)) {
        const unsigned tm = __builtin_amdgcn_readfirstlane(
            __hip_atomic_load((unsigned*)(myws + WS_TMO), __ATOMIC_RELAXED, __HIP_MEMORY_SCOPE_AGENT));
        if (tm == call || now - t0 > DEADLINE_TICKS) {
          __hip_atomic_store((unsigned*)(myws + WS_TMO), call, __ATOMIC_RELAXED, __HIP_MEMORY_SCOPE_AGENT);
          break;
        }
        __builtin_amdgcn_s_sleep(32);
      }
      const unsigned was = pend;
      asm volatile("s_waitcnt vmcnt(0)" : AF_OPS(0), AF_OPS(1), AF_OPS(2), AF_OPS(3) :: "memory");
#define CHECK(g) do { if (was & (1u << (g))) { const unsigned m_ = MING(g); DECIDE(g, m_); } } while (0)
      CHECK(0); CHECK(1); CHECK(2); CHECK(3);
      const unsigned ok = was & ~pend;
      if (ok & 1) MFMAG(0);
      if (ok & 2) MFMAG(1);
      if (ok & 4) MFMAG(2);
      if (ok & 8) MFMAG(3);
    }
    if (__builtin_expect(pend != 0, 0)) {
      asm volatile("s_waitcnt vmcnt(0)" : AF_OPS(0), AF_OPS(1), AF_OPS(2), AF_OPS(3) :: "memory");
      acc0[0] = __builtin_nanf(""), acc0 = acc0 + acc0[0], acc1 = acc1 + acc0[0];
    }
    STAMP(4);
  }

  blk = b;
  asm volatile("" : "+s"(blk));
  const unsigned reg = 2 * wv + (lane >> 5), eh = (lane >> 4) & 1;
  const unsigned ra = lds0 + ((lane >> 3) & 1) * 4096 + reg * 256 + eh * 128 + (lane & 7) * 16;
  const unsigned ovo = (bm * 32 + (reg & 3) + 8 * (reg >> 2) + 4 * eh) * (NDIM * 2) + bn * 128 + (lane & 15) * 8;
  const u32x4 srd_out = {ka[10], ka[11], MLOC * NDIM * 2, 0x00020000u};
  {
    const unsigned pa = lring + lane * 4;
#pragma unroll
    for (int r = 0; r < 16; ++r) {
      *(volatile __attribute__((address_space(3))) float*)(size_t)(pa + r * 256) = acc0[r];
      *(volatile __attribute__((address_space(3))) float*)(size_t)(pa + 4096 + r * 256) = acc1[r];
    }
    asm volatile("s_waitcnt lgkmcnt(0)\n\ts_barrier" :: "v"(ra), "v"(ovo), "s"(srd_out) : "memory");
  }
  STAMP(5);
  {
    f32x4 p[8];
    asm volatile(
        "ds_read_b128 %0, %8 offset:0\n\t"     "ds_read_b128 %1, %8 offset:20480\n\t"
        "ds_read_b128 %2, %8 offset:40960\n\t" "ds_read_b128 %3, %8 offset:61440\n\t"
        "ds_read_b128 %4, %9 offset:0\n\t"     "ds_read_b128 %5, %9 offset:20480\n\t"
        "ds_read_b128 %6, %9 offset:40960\n\t" "ds_read_b128 %7, %9 offset:61440\n\t"
        "s_waitcnt lgkmcnt(0)"
        : "=&v"(p[0]), "=&v"(p[1]), "=&v"(p[2]), "=&v"(p[3]), "=&v"(p[4]), "=&v"(p[5]), "=&v"(p[6]), "=&v"(p[7])
        : "v"(ra), "v"(ra + 4 * RS) : "memory");
    const f32x4 s = ((p[0] + p[1]) + (p[2] + p[3])) + ((p[4] + p[5]) + (p[6] + p[7]));
    bf16x2 o0 = {(__bf16)gelu_tanh(s[0]), (__bf16)gelu_tanh(s[1])};
    bf16x2 o1 = {(__bf16)gelu_tanh(s[2]), (__bf16)gelu_tanh(s[3])};
    u32x2 o = {__builtin_bit_cast(unsigned, o0), __builtin_bit_cast(unsigned, o1)};
    asm volatile("buffer_store_dwordx2 %0, %1, %2, 0 offen " OUT_AUX :: "v"(o), "v"(ovo), "s"(srd_out) : "memory");
    if (wv == 0) {
      asm volatile("buffer_store_dword %0, off, %1, %2" :: "v"(call), "s"(srd_ws), "s"(ctr_off) : "memory");
    }
  }
}

}
#undef rank
#undef srd_x
#undef srd_w
#undef myws

extern "C" void kernel_launch(int rank, int nranks, ncclComm_t comm,
                              hipStream_t stream,
                              void* const* d_in, const int* in_sizes, int n_in,
                              void* d_out, int out_size,
                              const int* in_dtypes, int out_dtype,
                              void* const* d_out_all,
                              void* const* d_ws_all, size_t ws_bytes) {
  if (nranks != NR || (unsigned)rank >= (unsigned)NR || n_in < 2 || ws_bytes < WS_NEED ||
      in_sizes[0] != 2048 * KLOC || in_sizes[1] != KDIM * NDIM || out_size != MLOC * NDIM) {
    fprintf(stderr, "unexpected problem shape\n");
    return;
  }
  auto srd = [](unsigned* d, const void* p, unsigned bytes) {
    const unsigned long long v = (unsigned long long)p;
    d[0] = (unsigned)v; d[1] = (unsigned)(v >> 32) & 0xffffu; d[2] = bytes; d[3] = 0x00020000u;
  };
  Args a = {};
  srd(a.srd_x, d_in[0], 2048u * KLOC * 2);
  srd(a.srd_w, (const char*)d_in[1] + (size_t)rank * (KLOC * NDIM * 2) - WBIAS, KLOC * NDIM * 2 + WBIAS);
  for (int v = 0; v < NR; ++v)
    srd(a.srd_wv[v], (const char*)d_in[1] + (size_t)((rank + v) & 7) * (KLOC * NDIM * 2) - WBIAS, KLOC * NDIM * 2 + WBIAS);
  a.myws = (unsigned long long)d_ws_all[rank];
  a.out = (unsigned long long)d_out;
  a.rank = (unsigned)rank;
  const unsigned part = WS_SLOT0 + (unsigned)rank * (MLOC * KLOC * 2);
  for (int p = 0; p < NR; ++p)
    srd(a.srd_dst[p], (const char*)d_ws_all[p] + part, (p == rank || NOSEND) ? 0u : (unsigned)WS_NEED - part);
  hipLaunchKernelGGL(a2a_gemm_gelu, dim3(NWG), dim3(512), 0, stream, a);
  HIP_CHECK(hipGetLastError());
}
```

```cpp
#include <hip/hip_runtime.h>
#include <rccl/rccl.h>
#include <cstdio>
#include <cstdint>
#pragma clang diagnostic ignored "-Winline-asm"

#define HIP_CHECK(expr) do { hipError_t _e = (expr); if (_e != hipSuccess) { \
    fprintf(stderr, "HIP error %s at %s:%d\n", hipGetErrorString(_e), __FILE__, __LINE__); \
    return; } } while (0)

#define X0 40
#define DQ 26
#define NH 14
#define GAP -10
#define OUTM 0
#define OUT_AUX ""
#define NOSEND 0
#define NOCHECK NOSEND
#define ORD 0,1,2,3,8,4,5,6,7,10,11,12,13
#define DEPTH 4
#define PRE 1
#define PRIO 1
#define ZMODE 5
#define ZAUX "sc0 sc1"
#define OWNT 0
#define XF 4
#define ROT 0
#define SPC 2
#define PRD 1
#define SLP 1
#define REV2 1
#define POLL1 1
#define ZT 5
#define PSL 0
#define CMAP 3
#define JMAP 1
#define KST 0

namespace {

constexpr int NR = 8;
constexpr int MLOC = 256, KDIM = 2048, NDIM = 2048, KLOC = 256;
constexpr int NWG = 256;
constexpr unsigned WS_CTR = 0;
constexpr unsigned WS_TMO = 2048;
constexpr unsigned WS_SLOT0 = 65536;
constexpr unsigned SLOT_BYTES = NR * MLOC * KLOC * 2;
constexpr unsigned NSLOT = 4;
constexpr unsigned WBIAS = 4096;
constexpr unsigned LB = 8192, L_OA = 15 * LB, L_FLAG = L_OA + 8 * 2048, L_END = L_FLAG + 1024;
constexpr unsigned slot_off(int j) { return j < 2 ? j * 4096u : 7 * LB + (j - 2) * 4096u; }
constexpr size_t WS_NEED = (size_t)WS_SLOT0 + (size_t)NSLOT * SLOT_BYTES;
#define SLOWT 100000ull
#define DLT 100000000ull
constexpr unsigned long long DEADLINE_TICKS = DLT;

typedef __attribute__((ext_vector_type(4))) unsigned u32x4;
typedef __attribute__((ext_vector_type(2))) unsigned u32x2;
typedef __attribute__((ext_vector_type(4))) float f32x4;
typedef __attribute__((ext_vector_type(16))) float f32x16;
typedef __attribute__((ext_vector_type(8))) __bf16 bf16x8;
typedef __attribute__((ext_vector_type(2))) __bf16 bf16x2;

typedef __attribute__((ext_vector_type(16))) unsigned u32x16;

struct Args {
  unsigned srd_x[4];
  unsigned srd_w[4];
  unsigned long long myws;
  unsigned long long out;
  unsigned rank, pad[3];
  unsigned srd_dst[NR][4];
  unsigned srd_wv[NR][4];
};
__device__ __forceinline__ unsigned umin3(unsigned a, unsigned b, unsigned c) {
  return __builtin_elementwise_min(__builtin_elementwise_min(a, b), c);
}
__device__ __forceinline__ unsigned min16(const u32x4& a, const u32x4& b, const u32x4& c, const u32x4& d) {
  unsigned m0 = umin3(a[0], a[1], a[2]), m1 = umin3(a[3], b[0], b[1]);
  m0 = umin3(m0, b[2], b[3]); m1 = umin3(m1, c[0], c[1]);
  m0 = umin3(m0, c[2], c[3]); m1 = umin3(m1, d[0], d[1]);
  m0 = umin3(m0, d[2], d[3]);
  return __builtin_elementwise_min(m0, m1);
}
__device__ __forceinline__ u32x2 gelu_tanh(const f32x4& y) {
  const f32x4 c1 = -2.30220819f, c3 = -0.10294324f;
  f32x4 t = y * __builtin_elementwise_fma(y * y, c3, c1);
  asm volatile("" : "+v"(t));
  for (int i = 0; i < 4; ++i) t[i] = __builtin_amdgcn_exp2f(t[i]);
  t += 1.0f;
  asm volatile("" : "+v"(t));
  for (int i = 0; i < 4; ++i) t[i] = __builtin_amdgcn_rcpf(t[i]);
  t *= y;
  const bf16x2 o0 = {(__bf16)t[0], (__bf16)t[1]}, o1 = {(__bf16)t[2], (__bf16)t[3]};
  return u32x2{__builtin_bit_cast(unsigned, o0), __builtin_bit_cast(unsigned, o1)};
}
__device__ __forceinline__ bf16x8 mk8(const u32x2& lo, const u32x2& hi) {
  u32x4 v = {lo[0], lo[1], hi[0], hi[1]};
  return __builtin_bit_cast(bf16x8, v);
}

#define DMA4(M0V, V, SRD, S0, S1, S2, S3) \
  asm volatile("s_mov_b32 m0, %0\n\ts_nop 0\n\t" \
               "buffer_load_dwordx4 %1, %2, %3 offen lds\n\t" \
               "buffer_load_dwordx4 %1, %2, %4 offen offset:1024 lds\n\t" \
               "buffer_load_dwordx4 %1, %2, %5 offen offset:2048 lds\n\t" \
               "buffer_load_dwordx4 %1, %2, %6 offen offset:3072 lds" \
               :: "s"(M0V), "v"(V), "s"(SRD), "s"(S0), "s"(S1), "s"(S2), "s"(S3) : "memory")

#define TR8(VM, dst, OFF) \
  asm volatile( \
    "s_waitcnt vmcnt(%c11)\n\t" \
    "ds_read_b64_tr_b16 %0, %8 offset:%c10+0\n\t"    "ds_read_b64_tr_b16 %1, %8 offset:%c10+512\n\t" \
    "ds_read_b64_tr_b16 %2, %9 offset:%c10+0\n\t"    "ds_read_b64_tr_b16 %3, %9 offset:%c10+512\n\t" \
    "ds_read_b64_tr_b16 %4, %8 offset:%c10+2048\n\t" "ds_read_b64_tr_b16 %5, %8 offset:%c10+2560\n\t" \
    "ds_read_b64_tr_b16 %6, %9 offset:%c10+2048\n\t" "ds_read_b64_tr_b16 %7, %9 offset:%c10+2560\n\t" \
    "s_waitcnt lgkmcnt(0)" \
    : "=&v"((dst)[0]), "=&v"((dst)[1]), "=&v"((dst)[2]), "=&v"((dst)[3]), "=&v"((dst)[4]), "=&v"((dst)[5]), "=&v"((dst)[6]), "=&v"((dst)[7]) \
    : "v"(ta0), "v"(ta1), "i"(OFF), "i"(VM) : "memory")

constexpr int SEQ[13] = {ORD};
constexpr int NIT = 13;
constexpr int item_cnt(int t) { return t >= NIT ? 0 : SEQ[t] == 8 ? 6 : 4; }
constexpr int ring_slot(int t) { int n = 0; for (int i = 0; i < t && i < NIT; ++i) n += SEQ[i] <= 8; return n % 4; }
constexpr int last_w() { int l = 0; for (int i = 0; i < NIT; ++i) if (SEQ[i] <= 8) l = i; return l; }
constexpr int issued_at(int t) { int n = t + DEPTH; return n > NIT ? NIT : n; }
static_assert(SEQ[NIT - 1] == 13, "group 3 comes last");
constexpr int vm_at(int t) { int n = 0; for (int i = t + 1; i < issued_at(t); ++i) n += item_cnt(i); return n; }
#define acc0 160
#define acc1 176
#define AFB 192
#define AQ_OPS(j, g) [r] "i"(AFB + 16 * (j)), [la] "v"(lane16), [srd] "s"(srd_ws), [q0] "s"(aq[g])
#define LOADG_TXT(R) \
    "buffer_load_dwordx4 v[" R ":" R "+3], %[la], %[srd], %[q0] offen sc0 sc1\n\t" \
    "buffer_load_dwordx4 v[" R "+4:" R "+7], %[la], %[srd], %[q0] offen offset:1024 sc0 sc1\n\t" \
    "buffer_load_dwordx4 v[" R "+8:" R "+11], %[la], %[srd], %[q0] offen offset:2048 sc0 sc1\n\t" \
    "buffer_load_dwordx4 v[" R "+12:" R "+15], %[la], %[srd], %[q0] offen offset:3072 sc0 sc1\n\t"
#define LOADG(j, g) asm volatile(LOADG_TXT("%c[r]") :: AQ_OPS(j, g) : "memory")
#define NOMOV(R, o) ""
#define MOV3(R, o) "v_mov_b64 v[240+" o ":241+" o "], v[" R "+" o ":" R "+1+" o "]\n\tv_mov_b64 v[242+" o ":243+" o "], v[" R "+2+" o ":" R "+3+" o "]\n\t"
#define ZERO_TXT(R, VM, MV) \
    "s_waitcnt vmcnt(" VM "+3)\n\t" \
    "v_min3_u32 %[t0], v[" R "], v[" R "+1], v[" R "+2]\n\t" MV(R, "0") \
    "s_waitcnt vmcnt(" VM "+2)\n\t" \
    "v_min3_u32 %[t1], v[" R "+3], v[" R "+4], v[" R "+5]\n\t" \
    "v_min3_u32 %[t0], %[t0], v[" R "+6], v[" R "+7]\n\t" MV(R, "4") \
    "s_waitcnt vmcnt(" VM "+1)\n\t" \
    "v_min3_u32 %[t1], %[t1], v[" R "+8], v[" R "+9]\n\t" \
    "v_min3_u32 %[t0], %[t0], v[" R "+10], v[" R "+11]\n\t" MV(R, "8") \
    "s_waitcnt vmcnt(" VM ")\n\t" \
    "v_min3_u32 %[t1], %[t1], v[" R "+12], v[" R "+13]\n\t" \
    "v_min3_u32 %[t0], %[t0], v[" R "+14], v[" R "+15]\n\t" \
    "v_min_u32_e32 %[t0], %[t0], %[t1]\n\t" \
    "v_cmp_gt_u32_e32 vcc, %c[one], %[t0]\n\t" MV(R, "12")
#define DECIDE(VM, g) do { unsigned t0_, t1_, ts_; asm volatile( \
    ZERO_TXT("%c[r]", "%c[vm]", NOMOV) \
    "s_cbranch_vccz 1f\n\t" \
    "s_and_b32 %[ts], %[pend], %c[low]\n\ts_cbranch_scc1 2f\n\t" \
    LOADG_TXT("%c[r]") \
    "s_branch 2f\n" \
    "1:\n\t" \
    "s_bitset0_b32 %[pend], %c[bit]\n" \
    "2:" \
    : [pend] "+s"(pend), [t0] "=&v"(t0_), [t1] "=&v"(t1_), [ts] "=&s"(ts_) \
    : [vm] "i"(VM), [bit] "i"(g), [low] "i"(POLL1 ? (1 << (g)) - 1 : 0), [one] "i"(NOCHECK ? 0 : 1), AQ_OPS(g, g) : "memory", "vcc", "scc"); } while (0)
#define ROT3(x) ""
#define ROTATE() do { unsigned t0_, t1_, n_ = 1024; asm volatile( \
    "s_sleep %c[spc]\n\t" LOADG_TXT("192") "s_sleep %c[spc]\n\t" LOADG_TXT("208") ROT3("s_sleep %c[spc]\n\t" LOADG_TXT("224")) \
    ZERO_TXT("240", "%c[vm]+4", NOMOV) \
    "s_cbranch_vccz 2f\n" \
    "1:\n\t" \
    ZERO_TXT("192", "%c[vm]", MOV3) "s_cbranch_vccz 2f\n\t" LOADG_TXT("192") \
    ZERO_TXT("208", "%c[vm]", MOV3) "s_cbranch_vccz 2f\n\t" LOADG_TXT("208") \
    ROT3(ZERO_TXT("224", "%c[vm]", MOV3) "s_cbranch_vccz 2f\n\t" LOADG_TXT("224")) \
    "s_sub_u32 %[n], %[n], 1\n\t" \
    "s_cbranch_scc0 1b\n\t" \
    "s_branch 3f\n" \
    "2:\n\t" \
    "s_mov_b32 %[pend], 0\n" \
    "3:" \
    : [pend] "+s"(pend), [t0] "=&v"(t0_), [t1] "=&v"(t1_), [n] "+s"(n_) \
    : [vm] "i"(4 * ROT - 4), [spc] "i"(SPC), [one] "i"(NOCHECK ? 0 : 1), AQ_OPS(3, 3) : "memory", "vcc", "scc"); \
    SWEEP((1024 - n_) * 100); } while (0)
#define MFA(acc, j, g, i, h) asm volatile("v_mfma_f32_32x32x16_bf16 v[%c0:%c0+15], v[%c2:%c2+3], %1, v[%c0:%c0+15]" :: "i"(acc), \
    "v"(mk8(wf[16 * (g) + 4 * (i) + 2 * (h)], wf[16 * (g) + 4 * (i) + 2 * (h) + 1])), "i"(AFB + 16 * (j) + 4 * (i)))
#define MFMAG(g) do { \
    MFA(acc0, g, g, 0, 0); MFA(acc1, g, g, 0, 1); MFA(acc0, g, g, 1, 0); MFA(acc1, g, g, 1, 1); \
    MFA(acc0, g, g, 2, 0); MFA(acc1, g, g, 2, 1); MFA(acc0, g, g, 3, 0); MFA(acc1, g, g, 3, 1); } while (0)
#define DW(acc, r, off) asm volatile("ds_write_addtid_b32 v[%c0] offset:%c1" :: "i"((acc) + (r)), "i"((off) + (r) * 256) : "memory")
#define DW2(acc, r, off) do { DW(acc, r, off); DW(acc, (r) + 1, off); } while (0)
#define DW8(acc, r, off) do { DW2(acc, r, off); DW2(acc, (r) + 2, off); DW2(acc, (r) + 4, off); DW2(acc, (r) + 6, off); } while (0)
#define NOP(n) asm volatile("s_nop %c0" :: "i"(n))

#define STAMP(i) do {} while (0)

__global__ __launch_bounds__(512, 2) __attribute__((amdgpu_num_vgpr(acc0 / 2))) void a2a_gemm_gelu(Args a) {
  __shared__ __attribute__((aligned(1024))) char lds[L_END];
  STAMP(0);
  const unsigned b = blockIdx.x;
  const void* const kp = (const void*)__builtin_amdgcn_kernarg_segment_ptr();
  u32x16 ka;
  const unsigned wv = __builtin_amdgcn_readfirstlane(threadIdx.x >> 6);
  const unsigned lds0 = (unsigned)(size_t)(__attribute__((address_space(3))) char*)lds;

  unsigned lane = threadIdx.x & 63, blk = b;
#define FENCE_IDS() asm volatile("" : "+v"(lane), "+s"(blk))
#define bn (((blk & 7) * 4 + (((blk >> 6) + CMAP) & 3)))
#define bm ((blk >> 3) & 7)
#define lane16 (lane * 16)
#define ctr_off (WS_CTR + ((blk & 7) * 32 + (blk >> 3)) * 4)
#define lring (lds0 + wv * LB)
#define loa (lds0 + L_OA + wv * 2048)
#define WL0 ((lane >> 3) * (NDIM * 2) + (((lane & 7) ^ ((lane >> 2) & 4)) * 16))
#define trow ((lane >> 5) * 8 + ((lane & 15) >> 2))
#define tch (((lane >> 4) & 1) * 2 + ((lane & 3) >> 1))
#define LATE_IDS() \
  const unsigned ta0 = lring + trow * 128 + ((tch ^ ((trow & 2) << 1)) * 16) + (lane & 1) * 8; \
  const unsigned ta1 = lring + trow * 128 + (((tch + 4) ^ ((trow & 2) << 1)) * 16) + (lane & 1) * 8; \
  const unsigned oar = loa + (lane & 31) * 64 + (lane >> 5) * 16
#define srd_x (u32x4{ka[0], ka[1], ka[2], ka[3]})
#define srd_w (u32x4{ka[4], ka[5], ka[6], ka[7]})
#define myws2 (u32x2{ka[8], ka[9]})
#define myws ((char*)(((unsigned long long)ka[9] << 32) | ka[8]))
#define rank (ka[12] & 7)
#define srd_ws (u32x4{ka[8], ka[9], (unsigned)WS_NEED, 0x00020000u})
#define oq ((bm + wv) & 7)
#define own_w (bn * 128 + WBIAS + oq * 131072u)
#define own_a ((rank * MLOC + bm * 32) * (KLOC * 2) + oq * 64)
#define oav ((lane >> 2) * (KLOC * 2) + (lane & 3) * 16)

  u32x2 wf[64];
  u32x2 ow[8];
  u32x4 oa[2];
  unsigned call, c0, f0;
  f32x4 p[8];
  unsigned ovo, sh0, fl_a;
#define share(t) (sh0 + (t) * LB)
#define hs ((wv >> 2) * 8)
#define TAIL_IDS() do { \
    ovo = (bm * 32 + (wv & 3) * 8 + (lane >> 4) + ((lane >> 1) & 4)) * (NDIM * 2) + bn * 128 + hs * 8 + (lane & 7) * 8; \
    sh0 = lds0 + wv * 1024 + lane16, fl_a = lds0 + L_FLAG; \
    asm volatile("" : "+v"(ovo), "+v"(sh0), "+v"(fl_a)); } while (0)
#define FLAG(m) asm volatile("s_mov_b64 exec, 1\n\tds_or_b32 %0, %1\n\ts_mov_b64 exec, -1" :: "v"(fl_a), "v"(m) : "memory")
#define LOAD_OA() asm volatile("s_mov_b32 m0, %0\n\ts_nop 0\n\t" \
    "buffer_load_dwordx4 %1, %3, %4 offen lds\n\tbuffer_load_dwordx4 %2, %3, %4 offen offset:1024 lds" \
    :: "s"(loa), "v"(oav), "v"(oav + 7168u), "s"(srd_x), "s"(own_a) : "memory")
#define READ_OA() asm volatile("ds_read_b128 %0, %2\n\tds_read_b128 %1, %2 offset:32\n\ts_waitcnt lgkmcnt(0)" \
    : "=&v"(oa[0]), "=&v"(oa[1]) : "v"(oar) : "memory")
#define MFMA_OWN() asm volatile( \
    "v_mfma_f32_32x32x16_bf16 v[160:175], %0, %2, 0\n\tv_mfma_f32_32x32x16_bf16 v[176:191], %0, %3, 0\n\t" \
    "v_mfma_f32_32x32x16_bf16 v[160:175], %1, %4, v[160:175]\n\tv_mfma_f32_32x32x16_bf16 v[176:191], %1, %5, v[176:191]" \
    :: "v"(oa[0]), "v"(oa[1]), "v"(mk8(ow[0], ow[1])), "v"(mk8(ow[2], ow[3])), "v"(mk8(ow[4], ow[5])), "v"(mk8(ow[6], ow[7])) : "v255")
#define DMA_OWN(slot) DMA4(lring + slot_off(slot), wl0, srd_w, own_w, own_w + 31744u, own_w + 2 * 31744u, own_w + 3 * 31744u)
#define WPOS(n) (((bm + ((n) >> 2)) & 7) * 131072u + bn * 128 + (WBIAS + ((n) & 3) * 31744u))
#define DMA_W(slot, it) DMA4(lring + slot_off(slot), wl0, srd_wv, WPOS(4 * (it)), WPOS(4 * (it) + 1), WPOS(4 * (it) + 2), WPOS(4 * (it) + 3))

#define ZERO4() do { \
      const unsigned z_off = WS_SLOT0 + ((call + 2) & (NSLOT - 1)) * SLOT_BYTES + b * 4096; \
      const u32x4 z = {0, 0, 0, 0}; \
      asm volatile("buffer_store_dwordx4 %0, %1, %2, %3 offen " ZAUX "\n\t" \
                   "buffer_store_dwordx4 %0, %1, %2, %3 offen offset:1024 " ZAUX "\n\t" \
                   "buffer_store_dwordx4 %0, %1, %2, %3 offen offset:2048 " ZAUX "\n\t" \
                   "buffer_store_dwordx4 %0, %1, %2, %3 offen offset:3072 " ZAUX "\n\ts_nop 1" \
                   :: "v"(z), "v"(lane16), "s"(srd_ws), "s"(z_off) : "memory"); } while (0)
  if (__builtin_expect(wv == 0, 1)) {
    asm volatile("s_load_dwordx16 %0, %3, 0x0\n\ts_setprio 3" : "=&s"(ka), "+s"(blk), "+v"(lane) : "s"(kp) : "memory");
    const unsigned job = blk < 64 ? blk >> 1 : blk - 32, slot = (job * 9363u) >> 16, sq = slot >> 3;
    const unsigned dd = (blk < 64 && !(blk & 1)) ? 8 : job - slot * 7 + 1;
    const unsigned sbm = (((slot & 7) ^ (REV2 && sq == 2 ? 7 : 0)) + dd) & 7;
    unsigned long long tstart;
    u32x4 srd_dst;
    const unsigned xo = (lane & 31) * (KLOC * 2) + (lane >> 5) * 16;
    const unsigned s_src = sbm * 32 * (KLOC * 2);
    unsigned s0 = s_src + ((sbm + 2 * sq) & 7) * 64, s1 = s_src + ((sbm + 2 * sq + 1) & 7) * 64;
    asm volatile("s_waitcnt lgkmcnt(0)" : "+s"(ka), "+s"(s0), "+s"(s1) : "v"(xo), "s"(dd) : "memory");
    const unsigned dest = (ka[12] + dd) & 7;
    s0 += dest * (MLOC * KLOC * 2);
    s1 += dest * (MLOC * KLOC * 2);
    if (KST) STAMP(3);
    u32x4 sx[4];
    asm volatile("buffer_load_dwordx4 %0, %4, %5, %6 offen\n\t"
                 "buffer_load_dwordx4 %1, %4, %5, %6 offen offset:32\n\t"
                 "buffer_load_dwordx4 %2, %4, %5, %7 offen\n\t"
                 "buffer_load_dwordx4 %3, %4, %5, %7 offen offset:32"
                 : "=&v"(sx[0]), "=&v"(sx[1]), "=&v"(sx[2]), "=&v"(sx[3])
                 : "v"(xo), "s"(srd_x), "s"(s0), "s"(s1) : "memory");
    FENCE_IDS();
    asm volatile("s_memrealtime %1\n\ts_load_dword %0, %3, %4\n\ts_load_dwordx4 %2, %5, %6"
                 : "=&s"(c0), "=&s"(tstart), "=&s"(srd_dst) : "s"(myws2), "s"(ctr_off), "s"(kp), "s"(0x40u + dest * 16u) : "memory");
    const unsigned wl0 = WL0;
    DMA_OWN(0);
    LOAD_OA();
    FENCE_IDS();
    const bool hold = slot < NH && dest != rank;
    if (!hold) asm volatile("s_barrier" ::: "memory");
    LATE_IDS();
    asm volatile("s_waitcnt lgkmcnt(0)" : "+s"(c0), "+s"(tstart), "+s"(srd_dst) :: "memory");
    call = c0 + 1;
    const unsigned d_off = sbm * 16384 + sq * 4096 + (call & (NSLOT - 1)) * SLOT_BYTES;
    const unsigned sv = lane16;
    const unsigned tgt = (unsigned)tstart + X0 + ((slot * DQ) >> 2) + (slot < NH ? 0 : GAP);
    asm volatile("s_waitcnt vmcnt(%c4)" : "+v"(sx[0]), "+v"(sx[1]), "+v"(sx[2]), "+v"(sx[3])
                 : "i"(OWNT == 0 ? 6 : 0), "s"(d_off), "s"(tgt), "v"(sv), "v"(ta0), "v"(ta1), "v"(oar) : "memory");
    if (__builtin_expect(__builtin_amdgcn_ballot_w64(min16(sx[0], sx[1], sx[2], sx[3]) == 0) != 0, 0)) {
#pragma unroll
      for (int i = 0; i < 4; ++i)
#pragma unroll
        for (int j = 0; j < 4; ++j) sx[i][j] = sx[i][j] ? sx[i][j] : 0x8000u;
    }
    STAMP(1);
    asm volatile("" :: "v"(sx[0]), "v"(sx[1]), "v"(sx[2]), "v"(sx[3]));
    while ((int)((unsigned)__builtin_amdgcn_s_memrealtime() - tgt) < 0) {}
    asm volatile("buffer_store_dwordx4 %0, %4, %5, %6 offen sc0 sc1\n\t"
                 "buffer_store_dwordx4 %1, %4, %5, %6 offen offset:1024 sc0 sc1\n\t"
                 "buffer_store_dwordx4 %2, %4, %5, %6 offen offset:2048 sc0 sc1\n\t"
                 "buffer_store_dwordx4 %3, %4, %5, %6 offen offset:3072 sc0 sc1\n\ts_nop 1"
                 :: "v"(sx[0]), "v"(sx[1]), "v"(sx[2]), "v"(sx[3]), "v"(sv), "s"(srd_dst), "s"(d_off) : "memory");
#define ZN 0
    if (hold) asm volatile("s_barrier" ::: "memory");
    __builtin_amdgcn_s_setprio(0);
    STAMP(2);
    TR8(OWNT == 3 ? 0 : 4 + ZN, ow, 0);
    READ_OA();
    MFMA_OWN();
    if (!KST) STAMP(3);
    STAMP(4);
    const unsigned fl_b = 0x101u;
    TAIL_IDS();
    asm volatile("s_mov_b32 m0, %0\n\ts_nop 11" :: "s"(lds0), "v"(fl_a), "v"(fl_b));
    DW8(acc0, 0, 0); DW8(acc0, 8, 0); DW8(acc1, 0, 4096); DW8(acc1, 8, 4096);
    FLAG(fl_b);
    { unsigned fs;
      asm volatile("1:\n\ts_sleep 2\n\tds_read_b32 %0, %2\n\ts_waitcnt lgkmcnt(0)\n\tv_readfirstlane_b32 %1, %0\n\t"
                   "s_and_b32 %1, %1, 0xff\n\ts_bcnt1_i32_b32 %1, %1\n\ts_cmp_lt_u32 %1, %c3\n\ts_cbranch_scc1 1b"
                   : "=&v"(f0), "=&s"(fs) : "v"(fl_a), "i"(ZT) : "memory", "scc"); }
    ZERO4();
    f0 = 0;
    asm volatile("" : "=v"(p[0]), "=v"(p[1]), "=v"(p[2]), "=v"(p[3]), "=v"(p[4]), "=v"(p[5]), "=v"(p[6]), "=v"(p[7]));
  } else {
    unsigned aq[4];
    unsigned pend = 15;
#define ISSUE(t) do { if constexpr ((t) < NIT) { constexpr int it_ = SEQ[(t) < NIT ? (t) : 0]; \
      if constexpr (it_ < 8) DMA_W(ring_slot(t), it_ & 7); \
      else if constexpr (it_ == 8) { DMA_OWN(ring_slot(t)); LOAD_OA(); } \
      else LOADG((it_ - 10) & 3, (it_ - 10) & 3); } } while (0)
#define ISSUE_RANGE(lo, hi) do { \
      if constexpr ((lo) < (hi)) ISSUE(lo); if constexpr ((lo) + 1 < (hi)) ISSUE((lo) + 1); \
      if constexpr ((lo) + 2 < (hi)) ISSUE((lo) + 2); if constexpr ((lo) + 3 < (hi)) ISSUE((lo) + 3); } while (0)
#define STEP(t) do { constexpr int it_ = SEQ[t]; \
      if constexpr (it_ < 8) { TR8(vm_at(t), wf + 8 * (it_ & 7), slot_off(ring_slot(t))); } \
      else if constexpr (it_ == 8) { TR8(vm_at(t), ow, slot_off(ring_slot(t))); } \
      if constexpr ((t) == 0) STAMP(1); \
      if constexpr ((t) == 0) { \
        asm volatile("" : "+s"(c0)); call = c0 + 1; \
        for (int g = 0; g < 4; ++g) aq[g] = WS_SLOT0 + (call & (NSLOT - 1)) * SLOT_BYTES + (src * 8 + bm) * 16384 + g * 4096; } \
      if constexpr (PRIO && (t) == 5) __builtin_amdgcn_s_setprio(PRIO - 1); \
      if constexpr (PRIO == 1 && (t) == 9) __builtin_amdgcn_s_setprio(1); \
      ISSUE_RANGE(issued_at(t), issued_at((t) + 1)); \
      if constexpr ((t) == last_w()) STAMP(2); \
      if constexpr (it_ == 8) { READ_OA(); MFMA_OWN(); } \
      if constexpr (it_ >= 10) { static_assert(it_ < 10 || issued_at(t) == NIT, "A groups come last"); \
        DECIDE(vm_at(t), (it_ - 10) & 3); \
        if constexpr (it_ < 13) { if (!(pend & (1u << ((it_ - 10) & 3)))) MFMAG((it_ - 10) & 3); } } } while (0)
    u32x4 srd_wv;
    asm volatile("s_load_dwordx4 %0, %2, %3\n\ts_load_dwordx16 %1, %2, 0x0" : "=&s"(srd_wv), "=&s"(ka) : "s"(kp), "s"(0xc0u + wv * 16u) : "memory");
    __builtin_amdgcn_s_setprio(2);
    const unsigned wl0 = WL0;
    asm volatile("s_waitcnt lgkmcnt(0)" : "+s"(ka), "+s"(srd_wv)
                 : "v"(wl0), "s"(lring), "s"(WPOS(0)), "s"(WPOS(1)), "s"(WPOS(2)), "s"(WPOS(3)) : "memory");
    __builtin_amdgcn_s_sleep(XF);
    ISSUE_RANGE(0, PRE);
    FENCE_IDS();
    asm volatile("ds_write_b32 %0, %1\n\ts_waitcnt lgkmcnt(0)" :: "v"(lds0 + L_FLAG), "v"(0u) : "memory");
    asm volatile("s_barrier\n\ts_load_dword %0, %1, %2" : "=&s"(c0) : "s"(myws2), "s"(ctr_off) : "memory");
    ISSUE_RANGE(PRE, DEPTH);
    FENCE_IDS();
    LATE_IDS();
    const unsigned src = (rank + wv) & 7;
    __builtin_amdgcn_s_setprio(PRIO);
    STEP(0); STEP(1); STEP(2); STEP(3); STEP(4); STEP(5); STEP(6); STEP(7); STEP(8);
    TAIL_IDS();
    unsigned fo = sh0 - lds0 + wv * LB, fl_1 = 1u << wv, fl_b = 0x100u << wv;
    asm volatile("" : "+v"(fo), "+v"(fl_1), "+v"(fl_b));
    STEP(9); STEP(10); STEP(11); STEP(12);
    STAMP(3);
    if (__builtin_expect(pend != 0, 0)) {
      unsigned long long t0 = 0;
#define SWEEP(n) do {} while (0)
#define EXPIRE() do { \
          const unsigned long long now = __builtin_amdgcn_s_memrealtime(); \
          if (t0 == 0) t0 = now; \
          if (__builtin_expect(now - t0 > SLOWT, 0)) { \
            const unsigned tm = __builtin_amdgcn_readfirstlane( \
                __hip_atomic_load((unsigned*)(myws + WS_TMO), __ATOMIC_RELAXED, __HIP_MEMORY_SCOPE_AGENT)); \
            if (tm == call || now - t0 > DEADLINE_TICKS) { \
              __hip_atomic_store((unsigned*)(myws + WS_TMO), call, __ATOMIC_RELAXED, __HIP_MEMORY_SCOPE_AGENT); \
              goto expired; \
            } \
            __builtin_amdgcn_s_sleep(32); \
          } } while (0)
#define CHECK(g) do { if (was & (1u << (g))) DECIDE(60, g); } while (0)
#define CHASE(g) do { if (pend & (1u << (g))) { \
          do { EXPIRE(); if (PSL) __builtin_amdgcn_s_sleep(PSL); DECIDE(0, g); SWEEP(1); } while (pend & (1u << (g))); \
          if ((g) < 1 && (pend & 2)) LOADG(1, 1); else if ((g) < 2 && (pend & 4)) LOADG(2, 2); else if (pend & 8) LOADG(3, 3); \
          STAMP((g) + 1); MFMAG(g); } } while (0)
      CHASE(0); CHASE(1); CHASE(2);
      while (!POLL1 && (pend & 7)) {
        EXPIRE();
        const unsigned was = pend;
        asm volatile("s_waitcnt vmcnt(0)" ::: "memory");
        CHECK(0); CHECK(1); CHECK(2); CHECK(3);
        const unsigned ok = was & ~pend;
        if (ok & 1) { STAMP(1); MFMAG(0); }
        if (ok & 2) { STAMP(2); MFMAG(1); }
        if (ok & 4) { STAMP(3); MFMAG(2); }
        SWEEP(1);
      }
      if (ROT && pend) ROTATE();
      while (__builtin_expect(pend != 0, !ROT)) {
        EXPIRE();
        DECIDE(0, 3);
        SWEEP(1);
      }
      goto done;
    expired: {
        const u32x4 nn = {0x7fc07fc0u, 0x7fc07fc0u, 0x7fc07fc0u, 0x7fc07fc0u};
        asm volatile("s_nop 15\n\tv_mfma_f32_32x32x16_bf16 v[160:175], %0, %0, v[160:175]\n\ts_nop 15" :: "v"(nn));
      }
    done:;
    }
    STAMP(4);
    MFA(acc0, 3, 3, 0, 0);
    asm volatile("s_mov_b32 m0, %1" : "+v"(fo) : "s"(lring));
#define READ3(F, fo) asm volatile(F "ds_read_b128 %1, %5\n\tds_read_b128 %2, %6\n\tds_read_b128 %3, %7" \
                   : "=&v"(f0), "=&v"(p[1]), "=&v"(p[2]), "=&v"(p[3]) \
                   : "v"(fl_a), "v"(lds0 + ((fo) ^ LB)), "v"(lds0 + ((fo) ^ (2 * LB))), "v"(lds0 + ((fo) ^ (3 * LB))) : "memory")
#define READ4(fo) asm volatile("ds_read_b128 %0, %4\n\tds_read_b128 %1, %5\n\tds_read_b128 %2, %6\n\tds_read_b128 %3, %7" \
                   : "=&v"(p[4]), "=&v"(p[5]), "=&v"(p[6]), "=&v"(p[7]) \
                   : "v"(lds0 + ((fo) ^ (4 * LB))), "v"(lds0 + ((fo) ^ (5 * LB))), "v"(lds0 + ((fo) ^ (6 * LB))), "v"(lds0 + ((fo) ^ (7 * LB))) : "memory")
    if (PRD) READ3("ds_read_b32 %0, %4\n\t", fo);
    MFA(acc0, 3, 3, 1, 0);
    if (PRD) READ4(fo);
    MFA(acc0, 3, 3, 2, 0);
    MFA(acc0, 3, 3, 3, 0);
    MFA(acc1, 3, 3, 0, 1); NOP(5);
    MFA(acc1, 3, 3, 1, 1); NOP(3); DW2(acc0, 0, 0);
    MFA(acc1, 3, 3, 2, 1); DW2(acc0, 2, 0); DW2(acc0, 4, 0); DW2(acc0, 6, 0);
    MFA(acc1, 3, 3, 3, 1);
    if (!PRD) f0 = 0;
    asm volatile("s_waitcnt lgkmcnt(8)" : "+v"(f0), "+v"(p[1]), "+v"(p[2]), "+v"(p[3]), "+v"(p[4]), "+v"(p[5]), "+v"(p[6]), "+v"(p[7]));
    DW8(acc0, 8, 0);
    FLAG(fl_1);
    DW8(acc1, 0, 4096); DW8(acc1, 8, 4096);
    FLAG(fl_b);
    if (PRD) asm volatile("ds_read_b128 %0, %2" : "=&v"(p[0]), "+v"(f0) : "v"(lds0 + fo) : "memory");
  }

  const u32x4 srd_out = {ka[10], ka[11], MLOC * NDIM * 2, 0x00020000u};
  const unsigned fsel = 0x80000u | hs;
  unsigned want = 0, fm = sh0;
  if (!PRD || __builtin_popcount(__builtin_amdgcn_ubfe(__builtin_amdgcn_readfirstlane(f0), hs, 8)) != 7) {
    unsigned fs, n;
    asm volatile("s_setprio 0\n"
                 "1:\n\tds_read_b32 %0, %3\n\ts_waitcnt lgkmcnt(0)\n\tv_readfirstlane_b32 %1, %0\n\t"
                 "s_bfe_u32 %1, %1, %4\n\ts_bcnt1_i32_b32 %2, %1\n\ts_cmp_gt_u32 %2, 6\n\ts_cbranch_scc1 2f\n\t"
                 "s_cmp_gt_u32 %2, 4\n\ts_cbranch_scc1 1b\n\ts_sleep %c5\n\ts_branch 1b\n"
                 "2:"
                 : "=&v"(f0), "=&s"(fs), "=&s"(n) : "v"(fl_a), "s"(fsel), "i"(SLP), "v"(p[0]) : "memory", "scc");
    const unsigned m = __builtin_ctz(~fs) & 7;
    const unsigned fo = sh0 - lds0 + m * LB;
    READ3("", fo);
    READ4(fo);
    fm = lds0 + fo;
    asm volatile("ds_read_b32 %0, %2\n\tds_read_b128 %1, %3" : "=&v"(f0), "=&v"(p[0]) : "v"(fl_a), "v"(fm) : "memory");
    want = (1u << m) << hs;
  }
  STAMP(5);
  asm volatile("s_waitcnt lgkmcnt(5)" : "+v"(p[1]), "+v"(p[2]), "+v"(p[3]), "+v"(p[4]) : "s"(srd_out));
  f32x4 sum = (p[1] + p[2]) + (p[3] + p[4]);
  asm volatile("s_waitcnt lgkmcnt(2)" : "+v"(p[5]), "+v"(p[6]), "+v"(p[7]), "+v"(sum));
  sum += (p[5] + p[6]) + p[7];
  asm volatile("s_waitcnt lgkmcnt(1)\n\tv_and_b32_e32 %0, %5, %0\n\tv_cmp_ne_u32_e32 vcc, %5, %0\n\ts_waitcnt lgkmcnt(0)\n\ts_cbranch_vccz 2f\n"
               "1:\n\tds_read_b32 %0, %3\n\tds_read_b128 %1, %4\n\ts_waitcnt lgkmcnt(1)\n\t"
               "v_and_b32_e32 %0, %5, %0\n\tv_cmp_ne_u32_e32 vcc, %5, %0\n\ts_waitcnt lgkmcnt(0)\n\ts_cbranch_vccnz 1b\n"
               "2:"
               : "+v"(f0), "+v"(p[0]), "+v"(sum) : "v"(fl_a), "v"(fm), "s"(want) : "memory", "vcc");
  sum += p[0];
  const u32x2 o = gelu_tanh(sum);
  asm volatile("buffer_store_dwordx2 %0, %1, %2, 0 offen " OUT_AUX :: "v"(o), "v"(ovo), "s"(srd_out) : "memory");
  if (wv == 0) {
    blk = b;
    asm volatile("" : "+s"(blk));
    call = __builtin_amdgcn_readfirstlane(call);
    asm volatile("buffer_store_dword %0, off, %1, %2" :: "v"(call), "s"(srd_ws), "s"(ctr_off) : "memory");
  }
}

}
#undef rank
#undef srd_x
#undef srd_w
#undef myws

extern "C" void kernel_launch(int rank, int nranks, ncclComm_t comm,
                              hipStream_t stream,
                              void* const* d_in, const int* in_sizes, int n_in,
                              void* d_out, int out_size,
                              const int* in_dtypes, int out_dtype,
                              void* const* d_out_all,
                              void* const* d_ws_all, size_t ws_bytes) {
  if (nranks != NR || (unsigned)rank >= (unsigned)NR || n_in < 2 || ws_bytes < WS_NEED ||
      in_sizes[0] != 2048 * KLOC || in_sizes[1] != KDIM * NDIM || out_size != MLOC * NDIM) {
    fprintf(stderr, "unexpected problem shape\n");
    return;
  }
  auto srd = [](unsigned* d, const void* p, unsigned bytes) {
    const unsigned long long v = (unsigned long long)p;
    d[0] = (unsigned)v; d[1] = (unsigned)(v >> 32) & 0xffffu; d[2] = bytes; d[3] = 0x00020000u;
  };
  Args a = {};
  srd(a.srd_x, d_in[0], 2048u * KLOC * 2);
  srd(a.srd_w, (const char*)d_in[1] + (size_t)rank * (KLOC * NDIM * 2) - WBIAS, KLOC * NDIM * 2 + WBIAS);
  for (int v = 0; v < NR; ++v)
    srd(a.srd_wv[v], (const char*)d_in[1] + (size_t)((rank + v) & 7) * (KLOC * NDIM * 2) - WBIAS, KLOC * NDIM * 2 + WBIAS);
  a.myws = (unsigned long long)d_ws_all[rank];
  a.out = (unsigned long long)d_out;
  a.rank = (unsigned)rank;
  const unsigned part = WS_SLOT0 + (unsigned)rank * (MLOC * KLOC * 2);
  for (int p = 0; p < NR; ++p)
    srd(a.srd_dst[p], (const char*)d_ws_all[p] + part, (p == rank || NOSEND) ? 0u : (unsigned)WS_NEED - part);
  hipLaunchKernelGGL(a2a_gemm_gelu, dim3(NWG), dim3(512), 0, stream, a);
  HIP_CHECK(hipGetLastError());
}
```
